```python
import numpy as np

kernel_name = "gpu_a2a_gemm_m4096_k4096_n16384_bf16_q8"


ABSMAX_RTOL = 0.02

COMPUTE_DTYPE = "bfloat16"
EPILOGUE = "quant_int8"
IN_QUANT = None
OUT_QUANT = "int8"
IN_DTYPES = {"x": "bfloat16", "w_mat": "bfloat16"}
WORLD_SIZE = 8

def _to_bf16(x):
    u = np.ascontiguousarray(x, dtype=np.float32).view(np.uint32)
    return ((u + 0x7FFF + ((u >> 16) & 1)) & 0xFFFF0000).view(np.float32)

def setup_inputs():

    x = _to_bf16(np.random.randn(4096, 4096).astype(np.float32))
    w_mat = _to_bf16((np.random.randn(4096, 16384) / np.sqrt(4096)).astype(np.float32))
    return {"x": x, "w_mat": w_mat}

def _quant_dequant(y, amax):
    scale = amax / 127.0
    q = np.clip(np.round(y / scale), -127.0, 127.0)
    return (q * scale).astype(np.float32)
def _epilogue(y):
    return _quant_dequant(y, np.abs(y).max())

def reference(x, w_mat):
    y = (x @ w_mat).astype(np.float32)
    return _epilogue(y)
```

```cpp
#include <hip/hip_runtime.h>
#include <rccl/rccl.h>
#include <cstdio>

#define HIP_CHECK(expr) do { hipError_t _e = (expr); if (_e != hipSuccess) { \
    fprintf(stderr, "HIP error %s at %s:%d\n", hipGetErrorString(_e), __FILE__, __LINE__); \
    return; } } while (0)

#define LAS __attribute__((address_space(3)))
typedef short bf16x8 __attribute__((ext_vector_type(8)));
typedef short s16x4 __attribute__((ext_vector_type(4)));
typedef float f32x4 __attribute__((ext_vector_type(4)));
typedef unsigned u32x4 __attribute__((ext_vector_type(4)));
typedef unsigned long long u64;

namespace {

constexpr int NR = 8;
constexpr int MLOC = 512, KTOT = 4096, NTOT = 16384;
constexpr int BM = 256, BN = 128, BK = 64, NKT = KTOT / BK;
constexpr int NWG = 256;
constexpr int A_STAGE = BM * BK * 2;
constexpr int B_STAGE = BK * BN * 2;
constexpr int NSA = 3, NSB = 3;
constexpr int LDS_A = 0, LDS_B = NSA * A_STAGE;
constexpr int LDS_MISC = LDS_B + NSB * B_STAGE;
constexpr int LDS_BYTES = LDS_MISC + 256;

constexpr size_t CHUNK_BYTES = (size_t)MLOC * 512 * 2;
constexpr size_t WS_STG = 0;
constexpr size_t WS_XF = WS_STG + NR * CHUNK_BYTES;
constexpr int XF_REPS = 4;
constexpr size_t WS_XF_BYTES = (size_t)XF_REPS * 2 * NR * 128;
constexpr size_t WS_LG = WS_XF + WS_XF_BYTES;
constexpr size_t WS_FG = WS_LG + NWG * 8;
constexpr int FG_REPS = 8;
constexpr size_t WS_CC = WS_FG + FG_REPS * 128;
constexpr size_t WS_TMO = WS_CC + NWG * 4;
constexpr size_t WS_END = WS_TMO + 128;

constexpr u64 DEADLINE_TICKS = 50000000ull;

struct Params {
  const char* x;
  const char* W;
  signed char* out;
  char* ws[NR];
  int rank;
};

#define BAR_LGKM() asm volatile("s_waitcnt lgkmcnt(0)\n\ts_barrier" ::: "memory")
#define BAR_ONLY() asm volatile("s_barrier" ::: "memory")
#define WAIT_VM(n) asm volatile("s_waitcnt vmcnt(" #n ")" ::: "memory")
#define SCHED_FENCE() __builtin_amdgcn_sched_barrier(0)

__device__ __forceinline__ void kt_decode(int j, int rank, int& p, int& s) {
  if (j < 8) { p = rank; s = j; }
  else { const int jj = j - 8; s = jj / 7; const int pi = jj - s * 7; p = (rank + 1 + pi) & 7; }
}
constexpr int STG_TILE1 = 8, STG_TILE2 = 8 + 7 * 1, STG_TILE3 = 8 + 7 * 4;

__device__ __forceinline__ u64 ld_sys64(const char* p) { return __hip_atomic_load((const u64*)p, __ATOMIC_RELAXED, __HIP_MEMORY_SCOPE_SYSTEM); }
__device__ __forceinline__ void st_sys64(char* p, u64 v) { __hip_atomic_store((u64*)p, v, __ATOMIC_RELAXED, __HIP_MEMORY_SCOPE_SYSTEM); }
__device__ __forceinline__ unsigned ld_agent32(const char* p) { return __hip_atomic_load((const unsigned*)p, __ATOMIC_RELAXED, __HIP_MEMORY_SCOPE_AGENT); }
__device__ __forceinline__ void st_agent32(char* p, unsigned v) { __hip_atomic_store((unsigned*)p, v, __ATOMIC_RELAXED, __HIP_MEMORY_SCOPE_AGENT); }

__device__ __forceinline__ bool give_up(char* myws, unsigned call, u64 t0) {
  const unsigned tm = __builtin_amdgcn_readfirstlane(ld_agent32(myws + WS_TMO));
  if (tm == call) return true;
  if (__builtin_amdgcn_s_memrealtime() - t0 > DEADLINE_TICKS) { st_agent32(myws + WS_TMO, call); return true; }
  return false;
}

#define STAMP(i) do { } while (0)

__global__ __launch_bounds__(512, 2) void a2a_gemm_q8(Params P) {
  extern __shared__ __attribute__((aligned(16))) char smem[];
  LAS char* lds = (LAS char*)smem;
  const int tid = threadIdx.x;
  const int wid = __builtin_amdgcn_readfirstlane(tid >> 6);
  const int lane = tid & 63;
  const int b = blockIdx.x;
  const int xcd = b & 7, idx = b >> 3;
  const int mt = idx & 1, nt = xcd * 16 + (idx >> 1);
  const int rank = P.rank;
  char* const myws = P.ws[rank];

  if (wid >= 4) {
    if (wid < 6) {
      const int h = wid - 4;
      const int r8 = lane >> 3, c8 = lane & 7;
      const unsigned va0 = (unsigned)(r8 * 1024 + ((c8 ^ ((0 + (lane >> 4)) & 7)) * 16));
      const unsigned va1 = (unsigned)(r8 * 1024 + ((c8 ^ ((4 + (lane >> 4)) & 7)) * 16));
      const unsigned rowbase = (unsigned)((mt * 256 + h * 128) * 1024);
      const int g4 = lane >> 4, c16 = lane & 15;
      const unsigned vw0 = (unsigned)(g4 * 32768 + ((c16 ^ ((g4 << 2) | 0)) * 16));
      const unsigned vw1 = (unsigned)(g4 * 32768 + ((c16 ^ ((g4 << 2) | 2)) * 16));
      auto wrsrc = __builtin_amdgcn_make_buffer_rsrc((void*)(P.W + nt * 256), 0, (int)((size_t)KTOT * NTOT * 2 - (size_t)nt * 256), 0x00020000);
      const char* const xloc = P.x + (size_t)rank * CHUNK_BYTES;
      auto issueA = [&](int j) {
        int p, s; kt_decode(j, rank, p, s);
        const char* base = (j < 8) ? xloc : (myws + WS_STG + (size_t)p * CHUNK_BYTES);
        auto rsrc = __builtin_amdgcn_make_buffer_rsrc((void*)base, 0, (int)CHUNK_BYTES, 0x00020000);
        LAS char* dst = lds + LDS_A + (j % NSA) * A_STAGE + h * 16384;
        const unsigned so = rowbase + s * 128;
#pragma unroll
        for (int i = 0; i < 16; ++i)
          __builtin_amdgcn_raw_ptr_buffer_load_lds(rsrc, (LAS void*)(dst + i * 1024), 16, (i & 1) ? va1 : va0, so + i * 8192, 0, 0);
      };
      auto issueW = [&](int j) {
        int p, s; kt_decode(j, rank, p, s);
        const unsigned so = (unsigned)(p * 512 + s * 64) * 32768u + (unsigned)h * (8u * 131072u);
        LAS char* dst = lds + LDS_B + (j % NSB) * B_STAGE + h * 8192;
#pragma unroll
        for (int i = 0; i < 8; ++i)
          __builtin_amdgcn_raw_ptr_buffer_load_lds(wrsrc, (LAS void*)(dst + i * 1024), 16, (i & 2) ? vw1 : vw0, so + i * 131072, 0, 0);
      };
      issueW(0); issueA(0); issueW(1); issueA(1); issueW(2);
      WAIT_VM(24); BAR_ONLY();
      issueA(2);
      BAR_ONLY();
      issueW(3);
      for (int t = 0; t < NKT; ++t) {
        if (t + 3 < NKT) WAIT_VM(24); else if (t + 2 < NKT) WAIT_VM(16); else WAIT_VM(0);
        BAR_ONLY();
        if (t + 3 < NKT) issueA(t + 3);
        if (t + 4 < NKT) issueW(t + 4);
      }
      return;
    }
    const unsigned call = __builtin_amdgcn_readfirstlane(ld_agent32(myws + WS_CC + b * 4)) + 1u;
    if (wid == 6) {
      STAMP(0);
      const bool active = b < 7 * 32;
      const int pj = b % 7, pu = b / 7;
      const int dest = (rank + 1 + pj) & 7;
      if (active) {
        const char* src = P.x + (size_t)dest * CHUNK_BYTES + (size_t)(pu * 16 + (lane >> 3)) * 1024 + (lane & 7) * 16;
        u32x4 v[16];
#pragma unroll
        for (int i = 0; i < 16; ++i) v[i] = *(const u32x4*)(src + (i & 1) * 8192 + (i >> 1) * 128);
        auto drsrc = __builtin_amdgcn_make_buffer_rsrc((void*)(P.ws[dest] + WS_STG + (size_t)rank * CHUNK_BYTES), 0, (int)CHUNK_BYTES, 0x00020000);
        const unsigned voff = (unsigned)((pu * 16 + (lane >> 3)) * 1024 + (lane & 7) * 16);
#pragma unroll
        for (int i = 0; i < 16; ++i)
          __builtin_amdgcn_raw_buffer_store_b128(v[i], drsrc, voff + (i & 1) * 8192 + (i >> 1) * 128, 0, 17  );
      }
      STAMP(1);
      char* fl = P.ws[dest] + WS_XF + ((size_t)((lane & (XF_REPS - 1)) * 2 + (pu >> 4)) * NR + rank) * 128 + (pu & 15) * 4;
      BAR_ONLY(); BAR_ONLY();
      for (int t = 0; t < NKT; ++t) {
        if (active) {
          if (t == 2) { WAIT_VM(14); if (lane < XF_REPS) __hip_atomic_store((unsigned*)fl, call * 4u + 1u, __ATOMIC_RELAXED, __HIP_MEMORY_SCOPE_SYSTEM); STAMP(2); }
          if (t == 7) { WAIT_VM(9);  if (lane < XF_REPS) __hip_atomic_store((unsigned*)fl, call * 4u + 2u, __ATOMIC_RELAXED, __HIP_MEMORY_SCOPE_SYSTEM); STAMP(3); }
          if (t == 16) { WAIT_VM(2); if (lane < XF_REPS) __hip_atomic_store((unsigned*)fl, call * 4u + 3u, __ATOMIC_RELAXED, __HIP_MEMORY_SCOPE_SYSTEM); STAMP(4); }
        }
        BAR_ONLY();
      }
      return;
    }
    {
      __builtin_amdgcn_fence(__ATOMIC_ACQUIRE, "");
      asm volatile("s_waitcnt vmcnt(0)" ::: "memory");
      const int pi = lane >> 3;
      const int src = (rank + 1 + pi) & 7;
      const char* fl = myws + WS_XF + ((size_t)(((idx >> 1) & (XF_REPS - 1)) * 2 + mt) * NR + src) * 128 + (lane & 7) * 8;
      auto flags_ok = [&](u64 v, unsigned need) { return pi == 7 || ((unsigned)v >= need && (unsigned)(v >> 32) >= need); };
      BAR_ONLY(); BAR_ONLY();
      u64 pre = 0;
      for (int t = 0; t < NKT; ++t) {
        const int stage = (t + 3 == STG_TILE1) ? 1 : (t + 3 == STG_TILE2) ? 2 : (t + 3 == STG_TILE3) ? 3 : 0;
        if (stage) {
          const unsigned need = call * 4u + (unsigned)stage;
          if (!__all(flags_ok(pre, need))) {
            const u64 t0 = __builtin_amdgcn_s_memrealtime();
            for (;;) {
              const u64 v = ld_sys64(fl);
              if (__all(flags_ok(v, need))) break;
              if (give_up(myws, call, t0)) break;
              __builtin_amdgcn_s_sleep(1);
            }
          }
          STAMP(4 + stage);
        }
        if (t + 4 == STG_TILE1 || t + 4 == STG_TILE2 || t + 4 == STG_TILE3) pre = ld_sys64(fl);
        BAR_ONLY();
      }
      BAR_LGKM();
      const LAS float* wmx = (const LAS float*)(lds + LDS_MISC);
      const float m4 = fmaxf(fmaxf(wmx[0], wmx[1]), fmaxf(wmx[2], wmx[3]));
      STAMP(8);
      if (lane == 0) __hip_atomic_store((u64*)(myws + WS_LG + b * 8), ((u64)call << 32) | __float_as_uint(m4), __ATOMIC_RELAXED, __HIP_MEMORY_SCOPE_AGENT);
      if (b == 0) {
        const u64 t0 = __builtin_amdgcn_s_memrealtime();
        unsigned mx;
        for (;;) {
          bool ok = true; mx = 0;
#pragma unroll
          for (int i = 0; i < 4; ++i) {
            const u64 v = __hip_atomic_load((const u64*)(myws + WS_LG + (i * 64 + lane) * 8), __ATOMIC_RELAXED, __HIP_MEMORY_SCOPE_AGENT);
            ok = ok && (unsigned)(v >> 32) >= call; mx = max(mx, (unsigned)v);
          }
          if (__all(ok)) break;
          if (give_up(myws, call, t0)) break;
          __builtin_amdgcn_s_sleep(1);
        }
#pragma unroll
        for (int o = 32; o >= 1; o >>= 1) mx = max(mx, (unsigned)__shfl_xor((int)mx, o));
        st_sys64(P.ws[lane >> 3] + WS_FG + (lane & 7) * 128 + rank * 8, ((u64)call << 32) | mx);
      }
      unsigned gmx;
      {
        const char* fg = myws + WS_FG + xcd * 128 + (lane & 7) * 8;
        const u64 t0 = __builtin_amdgcn_s_memrealtime();
        for (;;) {
          const u64 v = ld_sys64(fg);
          gmx = (unsigned)v;
          if (__all((unsigned)(v >> 32) >= call)) break;
          if (give_up(myws, call, t0)) break;
          __builtin_amdgcn_s_sleep(1);
        }
#pragma unroll
        for (int o = 4; o >= 1; o >>= 1) gmx = max(gmx, (unsigned)__shfl_xor((int)gmx, o));
      }
      const unsigned tm = __builtin_amdgcn_readfirstlane(ld_agent32(myws + WS_TMO));
      if (lane == 0) {
        LAS unsigned* o = (LAS unsigned*)(lds + LDS_MISC + 64);
        const float amax = __uint_as_float(gmx);
        const float scale = amax / 127.0f;
        o[0] = __float_as_uint(1.0f / scale);
        o[1] = (tm == call) ? 1u : 0u;
        st_agent32(myws + WS_CC + b * 4, call);
      }
      STAMP(9);
      BAR_LGKM();
      return;
    }
  }

  const int wm = wid >> 1, wn = wid & 1;
  const int l15 = lane & 15, g = lane >> 4, q = l15 >> 2, pp = lane & 3;
  unsigned baseA[2], baseB[4];
#pragma unroll
  for (int ks = 0; ks < 2; ++ks) baseA[ks] = (unsigned)((wm * 128 + l15) * 128 + (((4 * ks + g) ^ (l15 >> 1)) * 16)) + LDS_A;
  const int swz = (q << 2) | ((g & 1) << 1);
#pragma unroll
  for (int c = 0; c < 4; ++c) baseB[c] = (unsigned)((8 * g + q) * 256 + (((8 * wn + 2 * c + (pp >> 1)) ^ swz) * 16) + 8 * (pp & 1)) + LDS_B;

  f32x4 acc[8][4];
#pragma unroll
  for (int m = 0; m < 8; ++m)
#pragma unroll
    for (int c = 0; c < 4; ++c) acc[m][c] = (f32x4){0.f, 0.f, 0.f, 0.f};

  __builtin_amdgcn_s_setprio(1);
  if (wid == 0) STAMP(10);
  BAR_ONLY();
  if (wid == 0) STAMP(11);
  {
    bf16x8 bx[4][2], by[4][2], a0[2], a1[2];
    auto rdB1 = [&](bf16x8& dst, unsigned sb, int c, int ks) {
      const s16x4 lo = __builtin_amdgcn_ds_read_tr16_b64_v4i16((LAS s16x4*)(lds + (baseB[c] + sb) + ks * 8192));
      const s16x4 hi = __builtin_amdgcn_ds_read_tr16_b64_v4i16((LAS s16x4*)(lds + (baseB[c] + sb) + ks * 8192 + 1024));
      dst = __builtin_shufflevector(lo, hi, 0, 1, 2, 3, 4, 5, 6, 7);
    };
    auto rdA1 = [&](bf16x8 (&dst)[2], unsigned sa, int m) {
      dst[0] = *(const LAS bf16x8*)(lds + (baseA[0] + sa) + m * 2048);
      dst[1] = *(const LAS bf16x8*)(lds + (baseA[1] + sa) + m * 2048);
    };
#pragma unroll
    for (int c = 0; c < 4; ++c) { rdB1(bx[c][0], 0, c, 0); rdB1(bx[c][1], 0, c, 1); }
    rdA1(a0, 0, 0); rdA1(a1, 0, 1);
    BAR_LGKM();
    unsigned sa = 0, sbn = B_STAGE;
    auto tile = [&](bf16x8 (&bc)[4][2], bf16x8 (&bn)[4][2]) {
      unsigned san = sa + A_STAGE; if (san == NSA * A_STAGE) san = 0;
      bf16x8 a[10][2];
      a[0][0] = a0[0]; a[0][1] = a0[1]; a[1][0] = a1[0]; a[1][1] = a1[1];
#pragma unroll
      for (int m = 0; m < 8; ++m) {
#pragma unroll
        for (int i = 0; i < 4; ++i) {
          const int c = (m & 1) ? 3 - i : i;
          acc[m][c] = __builtin_amdgcn_mfma_f32_16x16x32_bf16(bc[c][0], a[m][0], acc[m][c], 0, 0, 0);
          if (i == 0) { if (m + 2 < 8) rdA1(a[m + 2], sa, m + 2); else rdA1(a[m + 2], san, m - 6); }
          if (i == 1 && m < 4) rdB1(bn[m][0], sbn, m, 0);
          if (i == 2 && m < 4) rdB1(bn[m][1], sbn, m, 1);
          acc[m][c] = __builtin_amdgcn_mfma_f32_16x16x32_bf16(bc[c][1], a[m][1], acc[m][c], 0, 0, 0);
          if (i == 0 || ((i == 1 || i == 2) && m < 4)) {
            __builtin_amdgcn_sched_group_barrier(0x8, 1, 0); __builtin_amdgcn_sched_group_barrier(0x100, 1, 0);
            __builtin_amdgcn_sched_group_barrier(0x8, 1, 0); __builtin_amdgcn_sched_group_barrier(0x100, 1, 0);
          }
          SCHED_FENCE();
        }
        if (m == 5) { BAR_LGKM(); SCHED_FENCE(); }
      }
      a0[0] = a[8][0]; a0[1] = a[8][1]; a1[0] = a[9][0]; a1[1] = a[9][1];
      sa = san;
      sbn += B_STAGE; if (sbn == NSB * B_STAGE) sbn = 0;
    };
    for (int t = 0; t < NKT; t += 2) { tile(bx, by); tile(by, bx); }
  }
  __builtin_amdgcn_s_setprio(0);
  if (wid == 0) STAMP(12);

  float amax = 0.f;
#pragma unroll
  for (int m = 0; m < 8; ++m)
#pragma unroll
    for (int c = 0; c < 4; ++c)
#pragma unroll
      for (int r = 0; r < 4; ++r) amax = fmaxf(amax, fabsf(acc[m][c][r]));
#pragma unroll
  for (int o = 32; o >= 1; o >>= 1) amax = fmaxf(amax, __shfl_xor(amax, o));
  if (lane == 0) *(LAS float*)(lds + LDS_MISC + wid * 4) = amax;
  BAR_LGKM();
  BAR_ONLY();
  const float inv = *(const LAS float*)(lds + LDS_MISC + 64);
  const unsigned bad = *(const LAS unsigned*)(lds + LDS_MISC + 68);
  if (wid == 0) STAMP(13);
  signed char* orow = P.out + (size_t)(mt * 256 + wm * 128 + l15) * NTOT + nt * 128 + wn * 64 + g * 16;
#pragma unroll
  for (int m = 0; m < 8; ++m) {
    unsigned d[4];
#pragma unroll
    for (int c = 0; c < 4; ++c) {
      unsigned w = 0;
#pragma unroll
      for (int r = 0; r < 4; ++r) {
        float u = __builtin_fmaf(acc[m][c][r], inv, 128.0f);
        u = __builtin_amdgcn_fmed3f(u, 1.0f, 255.0f);
        w = __builtin_amdgcn_cvt_pk_u8_f32(u, r, w);
      }
      d[c] = w ^ 0x80808080u;
    }
    {
      auto s0 = __builtin_amdgcn_permlane32_swap(d[0], d[2], false, false);
      auto s1 = __builtin_amdgcn_permlane32_swap(d[1], d[3], false, false);
      d[0] = s0[0]; d[2] = s0[1]; d[1] = s1[0]; d[3] = s1[1];
      auto s2 = __builtin_amdgcn_permlane16_swap(d[0], d[1], false, false);
      auto s3 = __builtin_amdgcn_permlane16_swap(d[2], d[3], false, false);
      d[0] = s2[0]; d[1] = s2[1]; d[2] = s3[0]; d[3] = s3[1];
    }
    u32x4 v = {d[0], d[1], d[2], d[3]};
    if (bad) v = (u32x4){0x80808080u, 0x80808080u, 0x80808080u, 0x80808080u};
    *(u32x4*)(orow + (size_t)m * 16 * NTOT) = v;
  }
}

}

extern "C" void kernel_launch(int rank, int nranks, ncclComm_t comm,
                              hipStream_t stream,
                              void* const* d_in, const int* in_sizes, int n_in,
                              void* d_out, int out_size,
                              const int* in_dtypes, int out_dtype,
                              void* const* d_out_all,
                              void* const* d_ws_all, size_t ws_bytes) {
  (void)comm; (void)in_dtypes; (void)out_dtype; (void)d_out_all;
  if (nranks != NR || n_in != 2 || (unsigned)rank >= (unsigned)NR || ws_bytes < WS_END + 65536 ||
      in_sizes[0] != KTOT * 512 || in_sizes[1] != KTOT * NTOT || out_size != MLOC * NTOT) {
    fprintf(stderr, "unexpected problem shape\n");
    return;
  }
  static bool attr_set = false;
  if (!attr_set) {
    HIP_CHECK(hipFuncSetAttribute((const void*)a2a_gemm_q8, hipFuncAttributeMaxDynamicSharedMemorySize, LDS_BYTES));
    attr_set = true;
  }
  Params P;
  P.x = (const char*)d_in[0];
  P.W = (const char*)d_in[1];
  P.out = (signed char*)d_out;
  for (int p = 0; p < NR; ++p) P.ws[p] = (char*)d_ws_all[p];
  P.rank = rank;
  hipLaunchKernelGGL(a2a_gemm_q8, dim3(NWG), dim3(512), LDS_BYTES, stream, P);
  HIP_CHECK(hipGetLastError());
}
```

```cpp
#include <hip/hip_runtime.h>
#include <rccl/rccl.h>
#include <cstdio>

#define HIP_CHECK(expr) do { hipError_t _e = (expr); if (_e != hipSuccess) { \
    fprintf(stderr, "HIP error %s at %s:%d\n", hipGetErrorString(_e), __FILE__, __LINE__); \
    return; } } while (0)

#define LAS __attribute__((address_space(3)))
typedef short bf16x8 __attribute__((ext_vector_type(8)));
typedef short s16x4 __attribute__((ext_vector_type(4)));
typedef float f32x4 __attribute__((ext_vector_type(4)));
typedef unsigned u32x4 __attribute__((ext_vector_type(4)));
typedef unsigned long long u64;

namespace {

constexpr int NR = 8;
constexpr int MLOC = 512, KTOT = 4096, NTOT = 16384;
constexpr int BM = 256, BN = 128, BK = 64, NKT = KTOT / BK;
constexpr int NWG = 256;
constexpr int A_STAGE = BM * BK * 2;
constexpr int B_STAGE = BK * BN * 2;
constexpr int NSA = 3, NSB = 3;
constexpr int LDS_A = 0, LDS_B = NSA * A_STAGE;
constexpr int LDS_MISC = LDS_B + NSB * B_STAGE;
constexpr int LDS_BYTES = LDS_MISC + 256;

constexpr size_t CHUNK_BYTES = (size_t)MLOC * 512 * 2;
constexpr size_t WS_STG = 0;
constexpr size_t WS_XF = WS_STG + NR * CHUNK_BYTES;
constexpr int XF_REPS = 4;
constexpr size_t WS_XF_BYTES = (size_t)XF_REPS * 2 * NR * 128;
constexpr size_t WS_LG = WS_XF + WS_XF_BYTES;
constexpr size_t WS_FG = WS_LG + NWG * 8;
constexpr int FG_REPS = 8;
constexpr size_t WS_CC = WS_FG + FG_REPS * 128;
constexpr size_t WS_TMO = WS_CC + NWG * 4;
constexpr size_t WS_END = WS_TMO + 128;

constexpr u64 DEADLINE_TICKS = 50000000ull;

struct Params {
  const char* x;
  const char* W;
  signed char* out;
  char* ws[NR];
  int rank;
};

template <int N> struct IC { static constexpr int value = N; };

#define BAR_LGKM() asm volatile("s_waitcnt lgkmcnt(0)\n\ts_barrier" ::: "memory")
#define BAR_ONLY() asm volatile("s_barrier" ::: "memory")
#define WAIT_VM(n) asm volatile("s_waitcnt vmcnt(" #n ")" ::: "memory")
#define SCHED_FENCE() __builtin_amdgcn_sched_barrier(0)

__device__ __forceinline__ void kt_decode(int j, int rank, int& p, int& s) {
  if (j < 8) { p = rank; s = j; }
  else { const int jj = j - 8; s = jj / 7; const int pi = jj - s * 7; p = (rank + 1 + pi) & 7; }
}
#define PUSH_STEP 4
#define PUSH_LAG 3
#define PUSH_T0 2
#define EARLY_INV 0

__device__ __forceinline__ u64 ld_sys64(const char* p) { return __hip_atomic_load((const u64*)p, __ATOMIC_RELAXED, __HIP_MEMORY_SCOPE_SYSTEM); }
__device__ __forceinline__ void st_sys64(char* p, u64 v) { __hip_atomic_store((u64*)p, v, __ATOMIC_RELAXED, __HIP_MEMORY_SCOPE_SYSTEM); }
__device__ __forceinline__ unsigned ld_agent32(const char* p) { return __hip_atomic_load((const unsigned*)p, __ATOMIC_RELAXED, __HIP_MEMORY_SCOPE_AGENT); }
__device__ __forceinline__ void st_agent32(char* p, unsigned v) { __hip_atomic_store((unsigned*)p, v, __ATOMIC_RELAXED, __HIP_MEMORY_SCOPE_AGENT); }

__device__ __forceinline__ bool give_up(char* myws, unsigned call, u64 t0, unsigned& spins) {
  if ((++spins & 15u) != 0u) return false;
  const unsigned tm = __builtin_amdgcn_readfirstlane(ld_agent32(myws + WS_TMO));
  if (tm == call) return true;
  if (__builtin_amdgcn_s_memrealtime() - t0 > DEADLINE_TICKS) { st_agent32(myws + WS_TMO, call); return true; }
  return false;
}

#define STAMP(i) do { } while (0)

__global__ __launch_bounds__(512, 2) void a2a_gemm_q8(Params P) {
  extern __shared__ __attribute__((aligned(16))) char smem[];
  LAS char* lds = (LAS char*)smem;
  const int tid = threadIdx.x;
  const int wid = __builtin_amdgcn_readfirstlane(tid >> 6);
  const int lane = tid & 63;
  const int b = blockIdx.x;
  const int xcd = b & 7, idx = b >> 3;
  const int mt = idx & 1, nt = xcd * 16 + (idx >> 1);
  const int rank = P.rank;
  char* const myws = P.ws[rank];

  if (wid >= 4) {
    if (wid < 6) {
      const int h = wid - 4;
      const int r8 = lane >> 3, c8 = lane & 7;
      const unsigned va0 = (unsigned)(r8 * 1024 + ((c8 ^ ((0 + (lane >> 4)) & 7)) * 16));
      const unsigned va1 = (unsigned)(r8 * 1024 + ((c8 ^ ((4 + (lane >> 4)) & 7)) * 16));
      const unsigned rowbase = (unsigned)((mt * 256 + h * 128) * 1024);
      const int g4 = lane >> 4, c16 = lane & 15;
      const unsigned vw0 = (unsigned)(g4 * 32768 + ((c16 ^ ((g4 << 2) | 0)) * 16));
      const unsigned vw1 = (unsigned)(g4 * 32768 + ((c16 ^ ((g4 << 2) | 2)) * 16));
      auto wrsrc = __builtin_amdgcn_make_buffer_rsrc((void*)(P.W + nt * 256), 0, (int)((size_t)KTOT * NTOT * 2 - (size_t)nt * 256), 0x00020000);
      const char* const xloc = P.x + (size_t)rank * CHUNK_BYTES;
      auto issueA = [&](int j) {
        int p, s; kt_decode(j, rank, p, s);
        const char* base = (j < 8) ? xloc : (myws + WS_STG + (size_t)p * CHUNK_BYTES);
        auto rsrc = __builtin_amdgcn_make_buffer_rsrc((void*)base, 0, (int)CHUNK_BYTES, 0x00020000);
        LAS char* dst = lds + LDS_A + (j % NSA) * A_STAGE + h * 16384;
        const unsigned so = rowbase + s * 128;
#pragma unroll
        for (int i = 0; i < 16; ++i)
          __builtin_amdgcn_raw_ptr_buffer_load_lds(rsrc, (LAS void*)(dst + i * 1024), 16, (i & 1) ? va1 : va0, so + i * 8192, 0, 0);
      };
      auto issueW = [&](int j) {
        int p, s; kt_decode(j, rank, p, s);
        const unsigned so = (unsigned)(p * 512 + s * 64) * 32768u + (unsigned)h * (8u * 131072u);
        LAS char* dst = lds + LDS_B + (j % NSB) * B_STAGE + h * 8192;
#pragma unroll
        for (int i = 0; i < 8; ++i)
          __builtin_amdgcn_raw_ptr_buffer_load_lds(wrsrc, (LAS void*)(dst + i * 1024), 16, (i & 2) ? vw1 : vw0, so + i * 131072, 0, 0);
      };
      issueW(0); issueA(0); issueW(1); issueA(1); issueW(2);
      WAIT_VM(24); BAR_ONLY();
      issueA(2);
      BAR_ONLY();
      issueW(3);
      for (int t = 0; t < NKT; ++t) {
        if (t + 3 < NKT) WAIT_VM(24); else if (t + 2 < NKT) WAIT_VM(16); else WAIT_VM(0);
        BAR_ONLY();
        if (t + 3 < NKT) issueA(t + 3);
        if (t + 4 < NKT) issueW(t + 4);
      }
      return;
    }
    if (wid == 6) {
      const bool active = b < 7 * 32;
      const int pj = b % 7, pu = b / 7;
      const int dest = (rank + 1 + pj) & 7;
      u32x4 v[16];
      const char* src = P.x + (size_t)dest * CHUNK_BYTES + (size_t)((pu & 31) * 16 + (lane >> 3)) * 1024 + (lane & 7) * 16;
      v[0] = *(const u32x4*)(src); v[1] = *(const u32x4*)(src + 8192);
      const unsigned callv = ld_agent32(myws + WS_CC + b * 4);
      asm volatile("" ::: "memory");
#pragma unroll
      for (int i = 2; i < 16; ++i) v[i] = *(const u32x4*)(src + (i & 1) * 8192 + (i >> 1) * 128);
      asm volatile("" ::: "memory");
      const unsigned call = __builtin_amdgcn_readfirstlane(callv) + 1u;
      STAMP(0);
      auto drsrc = __builtin_amdgcn_make_buffer_rsrc((void*)(P.ws[dest] + WS_STG + (size_t)rank * CHUNK_BYTES), 0, (int)CHUNK_BYTES, 0x00020000);
      const unsigned voff = (unsigned)((pu * 16 + (lane >> 3)) * 1024 + (lane & 7) * 16);
      char* fl = P.ws[dest] + WS_XF + ((size_t)((lane & (XF_REPS - 1)) * 2 + ((pu >> 4) & 1)) * NR + rank) * 128 + (pu & 15) * 4;
      auto push = [&](auto lo, auto hi) {
#pragma unroll
        for (int i = decltype(lo)::value; i < decltype(hi)::value; ++i)
          __builtin_amdgcn_raw_buffer_store_b128(v[i], drsrc, voff + (i & 1) * 8192 + (i >> 1) * 128, 0, 17  );
      };
      auto flag = [&](unsigned nslices) {
        WAIT_VM(0);
        if (lane < XF_REPS) __hip_atomic_store((unsigned*)fl, call * 8u + nslices, __ATOMIC_RELAXED, __HIP_MEMORY_SCOPE_SYSTEM);
      };
      int t = 0;
      auto bars_until = [&](int T) { for (; t < T; ++t) BAR_ONLY(); };
      if (active) push(IC<0>{}, IC<2>{});
      STAMP(1);
      BAR_ONLY(); BAR_ONLY();
      if (active) {
        const int ph = pu & (PUSH_STEP - 1);
        bars_until(PUSH_T0); flag(1); STAMP(2);
#define PUSH_SLICE(S) bars_until(PUSH_STEP * (S) - 2 + ph); push(IC<2 * (S)>{}, IC<2 * (S) + 2>{}); bars_until(PUSH_STEP * (S) - 2 + ph + PUSH_LAG); flag((S) + 1);
        PUSH_SLICE(1) STAMP(3);
        PUSH_SLICE(2) PUSH_SLICE(3) PUSH_SLICE(4) PUSH_SLICE(5) PUSH_SLICE(6) PUSH_SLICE(7)
        STAMP(4);
#undef PUSH_SLICE
      }
      bars_until(NKT);
      return;
    }
    const unsigned call = __builtin_amdgcn_readfirstlane(ld_agent32(myws + WS_CC + b * 4)) + 1u;
    {
      const int pi = lane >> 3;
      const int src = (rank + 1 + pi) & 7;
      const char* fl = myws + WS_XF + ((size_t)(((idx >> 1) & (XF_REPS - 1)) * 2 + mt) * NR + src) * 128 + (lane & 7) * 8;
      auto flags_ok = [&](u64 v, unsigned need) { return pi == 7 || ((unsigned)v >= need && (unsigned)(v >> 32) >= need); };
      BAR_ONLY(); BAR_ONLY();
      u64 pre = 0;
      for (int t = 0; t < NKT; ++t) {
        const int j = t + 3 - 8;
        if (j >= 0 && j < 56 && j % 7 == 0) {
          const unsigned need = call * 8u + (unsigned)(j / 7) + 1u;
          if (!__all(flags_ok(pre, need))) {
            const u64 t0 = __builtin_amdgcn_s_memrealtime(); unsigned spins = 0;
            for (;;) {
              const u64 v = ld_sys64(fl);
              if (__all(flags_ok(v, need))) break;
              if (give_up(myws, call, t0, spins)) break;
              __builtin_amdgcn_s_sleep(1);
            }
          }
          if (j == 0) STAMP(5); if (j == 7) STAMP(6); if (j == 49) STAMP(7);
        }
        if (j + 1 >= 0 && j + 1 < 56 && (j + 1) % 7 == 0) pre = ld_sys64(fl);
        BAR_ONLY();
      }
      BAR_LGKM();
      const LAS float* wmx = (const LAS float*)(lds + LDS_MISC);
      const float m4 = fmaxf(fmaxf(wmx[0], wmx[1]), fmaxf(wmx[2], wmx[3]));
      STAMP(8);
      if (lane == 0) __hip_atomic_store((u64*)(myws + WS_LG + b * 8), ((u64)call << 32) | __float_as_uint(m4), __ATOMIC_RELAXED, __HIP_MEMORY_SCOPE_AGENT);
      if (b == 0) {
        const u64 t0 = __builtin_amdgcn_s_memrealtime(); unsigned spins = 0;
        unsigned mx;
        for (;;) {
          bool ok = true; mx = 0;
#pragma unroll
          for (int i = 0; i < 4; ++i) {
            const u64 v = __hip_atomic_load((const u64*)(myws + WS_LG + (i * 64 + lane) * 8), __ATOMIC_RELAXED, __HIP_MEMORY_SCOPE_AGENT);
            ok = ok && (unsigned)(v >> 32) >= call; mx = max(mx, (unsigned)v);
          }
          if (__all(ok)) break;
          if (give_up(myws, call, t0, spins)) break;
          __builtin_amdgcn_s_sleep(1);
        }
#pragma unroll
        for (int o = 32; o >= 1; o >>= 1) mx = max(mx, (unsigned)__shfl_xor((int)mx, o));
        st_sys64(P.ws[lane >> 3] + WS_FG + (lane & 7) * 128 + rank * 8, ((u64)call << 32) | mx);
      }
      unsigned gmx;
      {
        const char* fg = myws + WS_FG + xcd * 128 + (lane & 7) * 8;
        const u64 t0 = __builtin_amdgcn_s_memrealtime(); unsigned spins = 0;
        for (;;) {
          const u64 v = ld_sys64(fg);
          gmx = (unsigned)v;
          if (__all((unsigned)(v >> 32) >= call)) break;
          if (give_up(myws, call, t0, spins)) break;
          __builtin_amdgcn_s_sleep(1);
        }
#pragma unroll
        for (int o = 4; o >= 1; o >>= 1) gmx = max(gmx, (unsigned)__shfl_xor((int)gmx, o));
      }
      const unsigned tm = __builtin_amdgcn_readfirstlane(ld_agent32(myws + WS_TMO));
      if (lane == 0) {
        LAS unsigned* o = (LAS unsigned*)(lds + LDS_MISC + 64);
        const float amax = __uint_as_float(gmx);
        const float scale = amax / 127.0f;
        o[0] = __float_as_uint(1.0f / scale);
        o[1] = (tm == call) ? 1u : 0u;
        st_agent32(myws + WS_CC + b * 4, call);
      }
      STAMP(9);
      BAR_LGKM();
      return;
    }
  }

  const int wm = wid >> 1, wn = wid & 1;
  const int l15 = lane & 15, g = lane >> 4, q = l15 >> 2, pp = lane & 3;
  unsigned baseA[2], baseB[4];
#pragma unroll
  for (int ks = 0; ks < 2; ++ks) baseA[ks] = (unsigned)((wm * 128 + l15) * 128 + (((4 * ks + g) ^ (l15 >> 1)) * 16)) + LDS_A;
  const int swz = (q << 2) | ((g & 1) << 1);
#pragma unroll
  for (int c = 0; c < 4; ++c) baseB[c] = (unsigned)((8 * g + q) * 256 + (((8 * wn + 2 * c + (pp >> 1)) ^ swz) * 16) + 8 * (pp & 1)) + LDS_B;

  f32x4 acc[8][4];
#pragma unroll
  for (int m = 0; m < 8; ++m)
#pragma unroll
    for (int c = 0; c < 4; ++c) acc[m][c] = (f32x4){0.f, 0.f, 0.f, 0.f};

  __builtin_amdgcn_s_setprio(1);
  if (wid == 0) STAMP(10);
  BAR_ONLY();
  if (wid == 0) STAMP(11);
  {
    bf16x8 bx[4][2], by[4][2], a0[2], a1[2];
    auto rdB1 = [&](bf16x8& dst, unsigned sb, int c, int ks) {
      const s16x4 lo = __builtin_amdgcn_ds_read_tr16_b64_v4i16((LAS s16x4*)(lds + (baseB[c] + sb) + ks * 8192));
      const s16x4 hi = __builtin_amdgcn_ds_read_tr16_b64_v4i16((LAS s16x4*)(lds + (baseB[c] + sb) + ks * 8192 + 1024));
      dst = __builtin_shufflevector(lo, hi, 0, 1, 2, 3, 4, 5, 6, 7);
    };
    auto rdA1 = [&](bf16x8 (&dst)[2], unsigned sa, int m) {
      dst[0] = *(const LAS bf16x8*)(lds + (baseA[0] + sa) + m * 2048);
      dst[1] = *(const LAS bf16x8*)(lds + (baseA[1] + sa) + m * 2048);
    };
#pragma unroll
    for (int c = 0; c < 4; ++c) { rdB1(bx[c][0], 0, c, 0); rdB1(bx[c][1], 0, c, 1); }
    rdA1(a0, 0, 0); rdA1(a1, 0, 1);
    BAR_LGKM();
    unsigned sa = 0, sbn = B_STAGE;
    auto tile = [&](bf16x8 (&bc)[4][2], bf16x8 (&bn)[4][2]) {
      unsigned san = sa + A_STAGE; if (san == NSA * A_STAGE) san = 0;
      bf16x8 a[10][2];
      a[0][0] = a0[0]; a[0][1] = a0[1]; a[1][0] = a1[0]; a[1][1] = a1[1];
#pragma unroll
      for (int m = 0; m < 8; ++m) {
#pragma unroll
        for (int i = 0; i < 4; ++i) {
          const int c = (m & 1) ? 3 - i : i;
          acc[m][c] = __builtin_amdgcn_mfma_f32_16x16x32_bf16(bc[c][0], a[m][0], acc[m][c], 0, 0, 0);
          if (i == 0) { if (m + 2 < 8) rdA1(a[m + 2], sa, m + 2); else rdA1(a[m + 2], san, m - 6); }
          if (i == 1 && m < 4) rdB1(bn[m][0], sbn, m, 0);
          if (i == 2 && m < 4) rdB1(bn[m][1], sbn, m, 1);
          acc[m][c] = __builtin_amdgcn_mfma_f32_16x16x32_bf16(bc[c][1], a[m][1], acc[m][c], 0, 0, 0);
          if (i == 0 || ((i == 1 || i == 2) && m < 4)) {
            __builtin_amdgcn_sched_group_barrier(0x8, 1, 0); __builtin_amdgcn_sched_group_barrier(0x100, 1, 0);
            __builtin_amdgcn_sched_group_barrier(0x8, 1, 0); __builtin_amdgcn_sched_group_barrier(0x100, 1, 0);
          }
          SCHED_FENCE();
        }
        if (m == 5) { BAR_LGKM(); SCHED_FENCE(); }
      }
      a0[0] = a[8][0]; a0[1] = a[8][1]; a1[0] = a[9][0]; a1[1] = a[9][1];
      sa = san;
      sbn += B_STAGE; if (sbn == NSB * B_STAGE) sbn = 0;
    };
    for (int t = 0; t < NKT; t += 2) { tile(bx, by); tile(by, bx); }
  }
  __builtin_amdgcn_s_setprio(0);
  if (wid == 0) STAMP(12);

  float amax = 0.f;
#pragma unroll
  for (int m = 0; m < 8; ++m)
#pragma unroll
    for (int c = 0; c < 4; ++c)
#pragma unroll
      for (int r = 0; r < 4; ++r) amax = fmaxf(amax, fabsf(acc[m][c][r]));
#pragma unroll
  for (int o = 32; o >= 1; o >>= 1) amax = fmaxf(amax, __shfl_xor(amax, o));
  if (lane == 0) *(LAS float*)(lds + LDS_MISC + wid * 4) = amax;
  BAR_LGKM();
  BAR_ONLY();
  const float inv = *(const LAS float*)(lds + LDS_MISC + 64);
  const unsigned bad = *(const LAS unsigned*)(lds + LDS_MISC + 68);
  if (wid == 0) STAMP(13);
  signed char* orow = P.out + (size_t)(mt * 256 + wm * 128 + l15) * NTOT + nt * 128 + wn * 64 + g * 16;
#pragma unroll
  for (int m = 0; m < 8; ++m) {
    unsigned d[4];
#pragma unroll
    for (int c = 0; c < 4; ++c) {
      unsigned w = 0;
#pragma unroll
      for (int r = 0; r < 4; ++r) {
        float u = __builtin_fmaf(acc[m][c][r], inv, 128.0f);
        u = __builtin_amdgcn_fmed3f(u, 1.0f, 255.0f);
        w = __builtin_amdgcn_cvt_pk_u8_f32(u, r, w);
      }
      d[c] = w ^ 0x80808080u;
    }
    {
      auto s0 = __builtin_amdgcn_permlane32_swap(d[0], d[2], false, false);
      auto s1 = __builtin_amdgcn_permlane32_swap(d[1], d[3], false, false);
      d[0] = s0[0]; d[2] = s0[1]; d[1] = s1[0]; d[3] = s1[1];
      auto s2 = __builtin_amdgcn_permlane16_swap(d[0], d[1], false, false);
      auto s3 = __builtin_amdgcn_permlane16_swap(d[2], d[3], false, false);
      d[0] = s2[0]; d[1] = s2[1]; d[2] = s3[0]; d[3] = s3[1];
    }
    u32x4 v = {d[0], d[1], d[2], d[3]};
    if (bad) v = (u32x4){0x80808080u, 0x80808080u, 0x80808080u, 0x80808080u};
    *(u32x4*)(orow + (size_t)m * 16 * NTOT) = v;
  }
}

}

extern "C" void kernel_launch(int rank, int nranks, ncclComm_t comm,
                              hipStream_t stream,
                              void* const* d_in, const int* in_sizes, int n_in,
                              void* d_out, int out_size,
                              const int* in_dtypes, int out_dtype,
                              void* const* d_out_all,
                              void* const* d_ws_all, size_t ws_bytes) {
  (void)comm; (void)in_dtypes; (void)out_dtype; (void)d_out_all;
  if (nranks != NR || n_in != 2 || (unsigned)rank >= (unsigned)NR || ws_bytes < WS_END + 65536 ||
      in_sizes[0] != KTOT * 512 || in_sizes[1] != KTOT * NTOT || out_size != MLOC * NTOT) {
    fprintf(stderr, "unexpected problem shape\n");
    return;
  }
  HIP_CHECK(hipFuncSetAttribute((const void*)a2a_gemm_q8, hipFuncAttributeMaxDynamicSharedMemorySize, LDS_BYTES));
  Params P;
  P.x = (const char*)d_in[0];
  P.W = (const char*)d_in[1];
  P.out = (signed char*)d_out;
  for (int p = 0; p < NR; ++p) P.ws[p] = (char*)d_ws_all[p];
  P.rank = rank;
  hipLaunchKernelGGL(a2a_gemm_q8, dim3(NWG), dim3(512), LDS_BYTES, stream, P);
  HIP_CHECK(hipGetLastError());
}
```

```cpp
#include <hip/hip_runtime.h>
#include <rccl/rccl.h>
#include <cstdio>

#define HIP_CHECK(expr) do { hipError_t _e = (expr); if (_e != hipSuccess) { \
    fprintf(stderr, "HIP error %s at %s:%d\n", hipGetErrorString(_e), __FILE__, __LINE__); \
    return; } } while (0)

#define LAS __attribute__((address_space(3)))
typedef short bf16x8 __attribute__((ext_vector_type(8)));
typedef short s16x4 __attribute__((ext_vector_type(4)));
typedef float f32x4 __attribute__((ext_vector_type(4)));
typedef unsigned u32x4 __attribute__((ext_vector_type(4)));
typedef unsigned long long u64;

namespace {

constexpr int NR = 8;
constexpr int MLOC = 512, KTOT = 4096, NTOT = 16384;
constexpr int BM = 256, BN = 128, BK = 64, NKT = KTOT / BK;
constexpr int NWG = 256;
constexpr int A_STAGE = BM * BK * 2;
constexpr int B_STAGE = BK * BN * 2;
constexpr int NSA = 3, NSB = 3;
constexpr int LDS_A = 0, LDS_B = NSA * A_STAGE;
constexpr int LDS_MISC = LDS_B + NSB * B_STAGE;
constexpr int LDS_BYTES = LDS_MISC + 256;

constexpr size_t CHUNK_BYTES = (size_t)MLOC * 512 * 2;
constexpr size_t WS_STG = 0;
constexpr size_t WS_XF = WS_STG + NR * CHUNK_BYTES;
constexpr int XF_REPS = 4;
constexpr size_t WS_XF_BYTES = (size_t)XF_REPS * 2 * NR * 128;
constexpr size_t WS_LG = WS_XF + WS_XF_BYTES;
constexpr size_t WS_FG = WS_LG + NWG * 4 * 8;
constexpr int FG_REPS = 8;
constexpr size_t WS_CC = WS_FG + FG_REPS * 128;
constexpr size_t WS_TMO = WS_CC + NWG * 4;
constexpr size_t WS_END = WS_TMO + 128;

constexpr u64 DEADLINE_TICKS = 50000000ull;

struct Params {
  const char* x;
  const char* W;
  signed char* out;
  char* ws[NR];
  int rank;
};

template <int N> struct IC { static constexpr int value = N; };

#define BAR_LGKM() asm volatile("s_waitcnt lgkmcnt(0)\n\ts_barrier" ::: "memory")
#define BAR_ONLY() asm volatile("s_barrier" ::: "memory")
#define WAIT_VM(n) asm volatile("s_waitcnt vmcnt(" #n ")" ::: "memory")
#define SCHED_FENCE() __builtin_amdgcn_sched_barrier(0)

__device__ __forceinline__ void kt_decode(int j, int rank, int& p, int& s) {
  if (j < 8) { p = rank; s = j; }
  else { const int jj = j - 8; s = jj / 7; const int pi = jj - s * 7; p = (rank + 1 + pi) & 7; }
}
#define PUSH_STEP 4
#define PUSH_LAG 3
#define PUSH_T0 2
#define OUT_AUX 0
#define PRO_ORDER 0
#define PRO_AFIRST 1
#define EARLY_INV 0

__device__ __forceinline__ u64 ld_sys64(const char* p) { return __hip_atomic_load((const u64*)p, __ATOMIC_RELAXED, __HIP_MEMORY_SCOPE_SYSTEM); }
__device__ __forceinline__ void st_sys64(char* p, u64 v) { __hip_atomic_store((u64*)p, v, __ATOMIC_RELAXED, __HIP_MEMORY_SCOPE_SYSTEM); }
__device__ __forceinline__ unsigned ld_agent32(const char* p) { return __hip_atomic_load((const unsigned*)p, __ATOMIC_RELAXED, __HIP_MEMORY_SCOPE_AGENT); }
__device__ __forceinline__ void st_agent32(char* p, unsigned v) { __hip_atomic_store((unsigned*)p, v, __ATOMIC_RELAXED, __HIP_MEMORY_SCOPE_AGENT); }

__device__ __forceinline__ bool give_up(char* myws, unsigned call, u64 t0, unsigned& spins) {
  if ((++spins & 15u) != 0u) return false;
  const unsigned tm = __builtin_amdgcn_readfirstlane(ld_agent32(myws + WS_TMO));
  if (tm == call) return true;
  if (__builtin_amdgcn_s_memrealtime() - t0 > DEADLINE_TICKS) { st_agent32(myws + WS_TMO, call); return true; }
  return false;
}

__device__ __forceinline__ unsigned wave_umax(unsigned v) {
  v = max(v, (unsigned)__builtin_amdgcn_update_dpp(0, (int)v, 0x111, 0xf, 0xf, false));
  v = max(v, (unsigned)__builtin_amdgcn_update_dpp(0, (int)v, 0x112, 0xf, 0xf, false));
  v = max(v, (unsigned)__builtin_amdgcn_update_dpp(0, (int)v, 0x114, 0xf, 0xf, false));
  v = max(v, (unsigned)__builtin_amdgcn_update_dpp(0, (int)v, 0x118, 0xf, 0xf, false));
  v = max(v, (unsigned)__builtin_amdgcn_update_dpp(0, (int)v, 0x142, 0xa, 0xf, false));
  v = max(v, (unsigned)__builtin_amdgcn_update_dpp(0, (int)v, 0x143, 0xc, 0xf, false));
  return v;
}

#define STAMP(i) do { } while (0)

__global__ __launch_bounds__(512, 2) void a2a_gemm_q8(Params P) {
  extern __shared__ __attribute__((aligned(16))) char smem[];
  LAS char* lds = (LAS char*)smem;
  const int tid = threadIdx.x;
  const int wid = __builtin_amdgcn_readfirstlane(tid >> 6);
  const int lane = tid & 63;
  const int b = blockIdx.x;
  const int xcd = b & 7, idx = b >> 3;
  const int mt = idx & 1, nt = xcd * 16 + (idx >> 1);
  const int rank = P.rank;
  char* const myws = P.ws[rank];
  unsigned call = 0;

  if (wid >= 4) {
    if (wid < 6) {
      const int h = wid - 4;
      const int r8 = lane >> 3, c8 = lane & 7;
      const unsigned va0 = (unsigned)(r8 * 1024 + ((c8 ^ ((0 + (lane >> 4)) & 7)) * 16));
      const unsigned va1 = (unsigned)(r8 * 1024 + ((c8 ^ ((4 + (lane >> 4)) & 7)) * 16));
      const unsigned rowbase = (unsigned)((mt * 256 + h * 128) * 1024);
      const int g4 = lane >> 4, c16 = lane & 15;
      const unsigned vw0 = (unsigned)(g4 * 32768 + ((c16 ^ ((g4 << 2) | 0)) * 16));
      const unsigned vw1 = (unsigned)(g4 * 32768 + ((c16 ^ ((g4 << 2) | 2)) * 16));
      auto wrsrc = __builtin_amdgcn_make_buffer_rsrc((void*)(P.W + nt * 256), 0, (int)((size_t)KTOT * NTOT * 2 - (size_t)nt * 256), 0x00020000);
      const char* const xloc = P.x + (size_t)rank * CHUNK_BYTES;
      auto issueA = [&](int j) {
        int p, s; kt_decode(j, rank, p, s);
        const char* base = (j < 8) ? xloc : (myws + WS_STG + (size_t)p * CHUNK_BYTES);
        auto rsrc = __builtin_amdgcn_make_buffer_rsrc((void*)base, 0, (int)CHUNK_BYTES, 0x00020000);
        LAS char* dst = lds + LDS_A + (j % NSA) * A_STAGE + h * 16384;
        const unsigned so = rowbase + s * 128;
#pragma unroll
        for (int i = 0; i < 16; ++i)
          __builtin_amdgcn_raw_ptr_buffer_load_lds(rsrc, (LAS void*)(dst + i * 1024), 16, (i & 1) ? va1 : va0, so + i * 8192, 0, 0);
      };
      auto issueW = [&](int j) {
        int p, s; kt_decode(j, rank, p, s);
        const unsigned so = (unsigned)(p * 512 + s * 64) * 32768u + (unsigned)h * (8u * 131072u);
        LAS char* dst = lds + LDS_B + (j % NSB) * B_STAGE + h * 8192;
#pragma unroll
        for (int i = 0; i < 8; ++i)
          __builtin_amdgcn_raw_ptr_buffer_load_lds(wrsrc, (LAS void*)(dst + i * 1024), 16, (i & 2) ? vw1 : vw0, so + i * 131072, 0, 0);
      };
      issueA(0); issueW(0); issueW(1);
      WAIT_VM(8);
      BAR_ONLY();
      issueA(1); issueW(2);
      WAIT_VM(24); BAR_ONLY();
      issueA(2); issueW(3);
      for (int t = 0; t < NKT; ++t) {
        if (t + 3 < NKT) WAIT_VM(24); else if (t + 2 < NKT) WAIT_VM(16); else WAIT_VM(0);
        BAR_ONLY();
        if (t + 3 < NKT) issueA(t + 3);
        if (t + 4 < NKT) issueW(t + 4);
      }
    } else if (wid == 6) {
      const bool active = b < 7 * 32;
      const int pj = b % 7, pu = b / 7;
      const int dest = (rank + 1 + pj) & 7;
      u32x4 v[16];
      const char* src = P.x + (size_t)dest * CHUNK_BYTES + (size_t)((pu & 31) * 16 + (lane >> 3)) * 1024 + (lane & 7) * 16;
      v[0] = *(const u32x4*)(src); v[1] = *(const u32x4*)(src + 8192);
      const unsigned callv = ld_agent32(myws + WS_CC + b * 4);
      asm volatile("" ::: "memory");
#pragma unroll
      for (int i = 2; i < 16; ++i) v[i] = *(const u32x4*)(src + (i & 1) * 8192 + (i >> 1) * 128);
      asm volatile("" ::: "memory");
      const unsigned pcall = __builtin_amdgcn_readfirstlane(callv) + 1u;
      STAMP(0);
      auto drsrc = __builtin_amdgcn_make_buffer_rsrc((void*)(P.ws[dest] + WS_STG + (size_t)rank * CHUNK_BYTES), 0, (int)CHUNK_BYTES, 0x00020000);
      const unsigned voff = (unsigned)(((pu & 31) * 16 + (lane >> 3)) * 1024 + (lane & 7) * 16);
      char* fl = P.ws[dest] + WS_XF + ((size_t)((lane & (XF_REPS - 1)) * 2 + ((pu >> 4) & 1)) * NR + rank) * 128 + (pu & 15) * 4;
      auto push = [&](auto lo, auto hi) {
#pragma unroll
        for (int i = decltype(lo)::value; i < decltype(hi)::value; ++i)
          __builtin_amdgcn_raw_buffer_store_b128(v[i], drsrc, voff + (i & 1) * 8192 + (i >> 1) * 128, 0, 17  );
      };
      auto flag = [&](unsigned nslices) {
        WAIT_VM(0);
        if (lane < XF_REPS) __hip_atomic_store((unsigned*)fl, pcall * 8u + nslices, __ATOMIC_RELAXED, __HIP_MEMORY_SCOPE_SYSTEM);
      };
      int t = 0;
      auto bars_until = [&](int T) { for (; t < T; ++t) BAR_ONLY(); };
      if (active) push(IC<0>{}, IC<2>{});
      STAMP(1);
      BAR_ONLY(); BAR_ONLY();
      if (active) {
        const int ph = pu & (PUSH_STEP - 1);
        bars_until(PUSH_T0); flag(1); STAMP(2);
#define PUSH_SLICE(S) bars_until(PUSH_STEP * (S) - 2 + ph); push(IC<2 * (S)>{}, IC<2 * (S) + 2>{}); bars_until(PUSH_STEP * (S) - 2 + ph + PUSH_LAG); flag((S) + 1);
        PUSH_SLICE(1) STAMP(3);
        PUSH_SLICE(2) PUSH_SLICE(3) PUSH_SLICE(4) PUSH_SLICE(5) PUSH_SLICE(6) PUSH_SLICE(7)
        STAMP(4);
#undef PUSH_SLICE
      }
      bars_until(NKT);
    } else {
      call = __builtin_amdgcn_readfirstlane(ld_agent32(myws + WS_CC + b * 4)) + 1u;
      if (lane == 0) *(LAS unsigned*)(lds + LDS_MISC) = call;
      const int pi = lane >> 3;
      const int src = (rank + 1 + pi) & 7;
      const char* fl = myws + WS_XF + ((size_t)(((idx >> 1) & (XF_REPS - 1)) * 2 + mt) * NR + src) * 128 + (lane & 7) * 8;
      auto flags_ok = [&](u64 v, unsigned need) { return pi == 7 || ((unsigned)v >= need && (unsigned)(v >> 32) >= need); };
      BAR_LGKM(); BAR_ONLY();
      u64 pre = 0;
      for (int t = 0; t < NKT; ++t) {
        const int j = t + 3 - 8;
        if (j >= 0 && j < 56 && j % 7 == 0) {
          const unsigned need = call * 8u + (unsigned)(j / 7) + 1u;
          if (!__all(flags_ok(pre, need))) {
            const u64 t0 = __builtin_amdgcn_s_memrealtime(); unsigned spins = 0;
            for (;;) {
              const u64 v = ld_sys64(fl);
              if (__all(flags_ok(v, need))) break;
              if (give_up(myws, call, t0, spins)) break;
              __builtin_amdgcn_s_sleep(1);
            }
          }
          if (j == 0) STAMP(5); if (j == 7) STAMP(6); if (j == 49) STAMP(7);
        }
        if (j + 1 >= 0 && j + 1 < 56 && (j + 1) % 7 == 0) pre = ld_sys64(fl);
        BAR_ONLY();
      }
    }
  } else {
    const int wm = wid >> 1, wn = wid & 1;
    const int l15 = lane & 15, g = lane >> 4, q = l15 >> 2, pp = lane & 3;
    unsigned baseA[2], baseB[4];
#pragma unroll
    for (int ks = 0; ks < 2; ++ks) baseA[ks] = (unsigned)((wm * 128 + l15) * 128 + (((4 * ks + g) ^ (l15 >> 1)) * 16)) + LDS_A;
    const int swz = (q << 2) | ((g & 1) << 1);
#pragma unroll
    for (int c = 0; c < 4; ++c) baseB[c] = (unsigned)((8 * g + q) * 256 + (((8 * wn + 2 * c + (pp >> 1)) ^ swz) * 16) + 8 * (pp & 1)) + LDS_B;

    f32x4 acc[8][4];
#pragma unroll
    for (int m = 0; m < 8; ++m)
#pragma unroll
      for (int c = 0; c < 4; ++c) acc[m][c] = (f32x4){0.f, 0.f, 0.f, 0.f};

    __builtin_amdgcn_s_setprio(1);
    if (wid == 0) STAMP(10);
    BAR_ONLY();
    if (wid == 0) STAMP(11);
    {
      bf16x8 bx[4][2], by[4][2], a0[2], a1[2];
      auto rdB1 = [&](bf16x8& dst, unsigned sb, int c, int ks) {
        const s16x4 lo = __builtin_amdgcn_ds_read_tr16_b64_v4i16((LAS s16x4*)(lds + (baseB[c] + sb) + ks * 8192));
        const s16x4 hi = __builtin_amdgcn_ds_read_tr16_b64_v4i16((LAS s16x4*)(lds + (baseB[c] + sb) + ks * 8192 + 1024));
        dst = __builtin_shufflevector(lo, hi, 0, 1, 2, 3, 4, 5, 6, 7);
      };
      auto rdA1 = [&](bf16x8 (&dst)[2], unsigned sa, int m) {
        dst[0] = *(const LAS bf16x8*)(lds + (baseA[0] + sa) + m * 2048);
        dst[1] = *(const LAS bf16x8*)(lds + (baseA[1] + sa) + m * 2048);
      };
      call = __builtin_amdgcn_readfirstlane(*(const LAS unsigned*)(lds + LDS_MISC));
#pragma unroll
      for (int c = 0; c < 4; ++c) { rdB1(bx[c][0], 0, c, 0); rdB1(bx[c][1], 0, c, 1); }
      rdA1(a0, 0, 0); rdA1(a1, 0, 1);
      BAR_LGKM();
      unsigned sa = 0, sbn = B_STAGE;
      auto tile = [&](bf16x8 (&bc)[4][2], bf16x8 (&bn)[4][2]) {
        unsigned san = sa + A_STAGE; if (san == NSA * A_STAGE) san = 0;
        bf16x8 a[10][2];
        a[0][0] = a0[0]; a[0][1] = a0[1]; a[1][0] = a1[0]; a[1][1] = a1[1];
#pragma unroll
        for (int m = 0; m < 8; ++m) {
#pragma unroll
          for (int i = 0; i < 4; ++i) {
            const int c = (m & 1) ? 3 - i : i;
            acc[m][c] = __builtin_amdgcn_mfma_f32_16x16x32_bf16(bc[c][0], a[m][0], acc[m][c], 0, 0, 0);
            if (i == 0) { if (m + 2 < 8) rdA1(a[m + 2], sa, m + 2); else rdA1(a[m + 2], san, m - 6); }
            if (i == 1 && m < 4) rdB1(bn[m][0], sbn, m, 0);
            if (i == 2 && m < 4) rdB1(bn[m][1], sbn, m, 1);
            acc[m][c] = __builtin_amdgcn_mfma_f32_16x16x32_bf16(bc[c][1], a[m][1], acc[m][c], 0, 0, 0);
            if (i == 0 || ((i == 1 || i == 2) && m < 4)) {
              __builtin_amdgcn_sched_group_barrier(0x8, 1, 0); __builtin_amdgcn_sched_group_barrier(0x100, 1, 0);
              __builtin_amdgcn_sched_group_barrier(0x8, 1, 0); __builtin_amdgcn_sched_group_barrier(0x100, 1, 0);
            }
            SCHED_FENCE();
          }
          if (m == 5) { BAR_LGKM(); SCHED_FENCE(); }
        }
        a0[0] = a[8][0]; a0[1] = a[8][1]; a1[0] = a[9][0]; a1[1] = a[9][1];
        sa = san;
        sbn += B_STAGE; if (sbn == NSB * B_STAGE) sbn = 0;
      };
      for (int t = 0; t < NKT; t += 2) { tile(bx, by); tile(by, bx); }
    }
    __builtin_amdgcn_s_setprio(0);
    if (wid == 0) STAMP(12);
    float am0 = 0.f, am1 = 0.f;
#pragma unroll
    for (int m = 0; m < 8; ++m)
#pragma unroll
      for (int c = 0; c < 4; ++c) {
        am0 = fmaxf(fmaxf(am0, fabsf(acc[m][c][0])), fabsf(acc[m][c][1]));
        am1 = fmaxf(fmaxf(am1, fabsf(acc[m][c][2])), fabsf(acc[m][c][3]));
      }
    const unsigned wmax = wave_umax(__float_as_uint(fmaxf(am0, am1)));
    if (lane == 63) __hip_atomic_store((u64*)(myws + WS_LG + (b * 4 + wid) * 8), ((u64)call << 32) | wmax, __ATOMIC_RELAXED, __HIP_MEMORY_SCOPE_AGENT);
    if (wid == 0) STAMP(8);
#pragma unroll
    for (int m = 0; m < 8; ++m)
#pragma unroll
      for (int c = 0; c < 4; ++c) {
        const int row = wm * 128 + m * 16 + l15;
        *(LAS f32x4*)(lds + row * 512 + (((wn * 16 + c * 4 + g) ^ (l15 & 7)) * 16)) = acc[m][c];
      }
  }

  BAR_LGKM();
  f32x4 yv[4][4];
  {
    const int r3 = lane >> 3, s8 = lane & 7;
#pragma unroll
    for (int i = 0; i < 4; ++i)
#pragma unroll
      for (int j = 0; j < 4; ++j)
        yv[i][j] = *(const LAS f32x4*)(lds + (wid * 32 + i * 8 + r3) * 512 + (((s8 * 4 + j) ^ r3) * 16));
  }
  if (wid == 7) {
    if (b == 0) {
      const u64 t0 = __builtin_amdgcn_s_memrealtime(); unsigned spins = 0;
      unsigned mx;
      for (;;) {
        bool ok = true; mx = 0;
#pragma unroll
        for (int i = 0; i < 16; ++i) {
          const u64 v = __hip_atomic_load((const u64*)(myws + WS_LG + (i * 64 + lane) * 8), __ATOMIC_RELAXED, __HIP_MEMORY_SCOPE_AGENT);
          ok = ok && (unsigned)(v >> 32) >= call; mx = max(mx, (unsigned)v);
        }
        if (__all(ok)) break;
        if (give_up(myws, call, t0, spins)) break;
      }
      mx = wave_umax(mx);
      mx = __builtin_amdgcn_readlane(mx, 63);
      st_sys64(P.ws[lane >> 3] + WS_FG + (lane & 7) * 128 + rank * 8, ((u64)call << 32) | mx);
    }
    unsigned gmx;
    {
      const char* fg = myws + WS_FG + xcd * 128 + (lane & 7) * 8;
      const u64 t0 = __builtin_amdgcn_s_memrealtime(); unsigned spins = 0;
      for (;;) {
        const u64 v = ld_sys64(fg);
        gmx = (unsigned)v;
        if (__all((unsigned)(v >> 32) >= call)) break;
        if (give_up(myws, call, t0, spins)) break;
      }
      gmx = __builtin_amdgcn_readlane(wave_umax(gmx), 63);
    }
    const unsigned tm = __builtin_amdgcn_readfirstlane(ld_agent32(myws + WS_TMO));
    if (lane == 0) {
      LAS unsigned* o = (LAS unsigned*)(lds + LDS_MISC + 64);
      const float scale = __uint_as_float(gmx) / 127.0f;
      o[0] = __float_as_uint(1.0f / scale);
      o[1] = (tm == call) ? 1u : 0u;
      st_agent32(myws + WS_CC + b * 4, call);
    }
    STAMP(9);
  }
  BAR_LGKM();
  const float inv = *(const LAS float*)(lds + LDS_MISC + 64);
  const unsigned bad = *(const LAS unsigned*)(lds + LDS_MISC + 68);
  if (wid == 0) STAMP(13);
  {
    auto orsrc = __builtin_amdgcn_make_buffer_rsrc((void*)P.out, 0, MLOC * NTOT, 0x00020000);
    const unsigned ovoff = (unsigned)((mt * 256 + wid * 32 + (lane >> 3)) * NTOT + nt * 128 + (lane & 7) * 16);
#pragma unroll
    for (int i = 0; i < 4; ++i) {
      unsigned d[4];
#pragma unroll
      for (int j = 0; j < 4; ++j) {
        unsigned w = 0;
#pragma unroll
        for (int r = 0; r < 4; ++r) {
          float u = __builtin_fmaf(yv[i][j][r], inv, 128.0f);
          u = __builtin_amdgcn_fmed3f(u, 1.0f, 255.0f);
          w = __builtin_amdgcn_cvt_pk_u8_f32(u, r, w);
        }
        d[j] = bad ? 0x80808080u : (w ^ 0x80808080u);
      }
      const u32x4 v = {d[0], d[1], d[2], d[3]};
      __builtin_amdgcn_raw_buffer_store_b128(v, orsrc, ovoff + i * 8 * NTOT, 0, OUT_AUX);
    }
  }
}

}

extern "C" void kernel_launch(int rank, int nranks, ncclComm_t comm,
                              hipStream_t stream,
                              void* const* d_in, const int* in_sizes, int n_in,
                              void* d_out, int out_size,
                              const int* in_dtypes, int out_dtype,
                              void* const* d_out_all,
                              void* const* d_ws_all, size_t ws_bytes) {
  (void)comm; (void)in_dtypes; (void)out_dtype; (void)d_out_all;
  if (nranks != NR || n_in != 2 || (unsigned)rank >= (unsigned)NR || ws_bytes < WS_END + 131072 ||
      in_sizes[0] != KTOT * 512 || in_sizes[1] != KTOT * NTOT || out_size != MLOC * NTOT) {
    fprintf(stderr, "unexpected problem shape\n");
    return;
  }
  HIP_CHECK(hipFuncSetAttribute((const void*)a2a_gemm_q8, hipFuncAttributeMaxDynamicSharedMemorySize, LDS_BYTES));
  Params P;
  P.x = (const char*)d_in[0];
  P.W = (const char*)d_in[1];
  P.out = (signed char*)d_out;
  for (int p = 0; p < NR; ++p) P.ws[p] = (char*)d_ws_all[p];
  P.rank = rank;
  hipLaunchKernelGGL(a2a_gemm_q8, dim3(NWG), dim3(512), LDS_BYTES, stream, P);
  HIP_CHECK(hipGetLastError());
}
```

```cpp
#include <hip/hip_runtime.h>
#include <rccl/rccl.h>
#include <cstdio>

#define HIP_CHECK(expr) do { hipError_t _e = (expr); if (_e != hipSuccess) { \
    fprintf(stderr, "HIP error %s at %s:%d\n", hipGetErrorString(_e), __FILE__, __LINE__); \
    return; } } while (0)

#define LAS __attribute__((address_space(3)))
typedef short bf16x8 __attribute__((ext_vector_type(8)));
typedef short s16x4 __attribute__((ext_vector_type(4)));
typedef float f32x4 __attribute__((ext_vector_type(4)));
typedef unsigned u32x4 __attribute__((ext_vector_type(4)));
typedef unsigned long long u64;

namespace {

constexpr int NR = 8;
constexpr int MLOC = 512, KTOT = 4096, NTOT = 16384;
constexpr int BM = 256, BN = 128, BK = 64, NKT = KTOT / BK;
constexpr int NWG = 256;
constexpr int A_STAGE = BM * BK * 2;
constexpr int B_STAGE = BK * BN * 2;
#define NSB_ 3
constexpr int NSA = 3, NSB = NSB_;
constexpr int LDS_A = 0, LDS_B = NSA * A_STAGE;
constexpr int LDS_BYTES = LDS_B + NSB * B_STAGE;
constexpr int LDS_MISC = 131072;
static_assert(LDS_BYTES <= 163840, "LDS");

constexpr size_t CHUNK_BYTES = (size_t)MLOC * 512 * 2;
constexpr size_t WS_STG = 0;
constexpr size_t WS_XF = WS_STG + NR * CHUNK_BYTES;
constexpr int XF_REPS = 4;
constexpr size_t WS_XF_BYTES = (size_t)XF_REPS * 2 * NR * 128;
constexpr size_t WS_LG = WS_XF + WS_XF_BYTES;
constexpr size_t WS_FG = WS_LG + NWG * 4 * 8;
constexpr int FG_REPS = 8;
constexpr size_t WS_CC = WS_FG + FG_REPS * 128;
constexpr size_t WS_TMO = WS_CC + NWG * 4;
constexpr size_t WS_END = WS_TMO + 128;

constexpr u64 DEADLINE_TICKS = 50000000ull;

struct Params {
  const char* x;
  const char* W;
  signed char* out;
  char* ws[NR];
  int rank;
};

template <int N> struct IC { static constexpr int value = N; };

#define BAR_LGKM() asm volatile("s_waitcnt lgkmcnt(0)\n\ts_barrier" ::: "memory")
#define BAR_ONLY() asm volatile("s_barrier" ::: "memory")
#define WAIT_VM(n) asm volatile("s_waitcnt vmcnt(" #n ")" ::: "memory")
#define SCHED_FENCE() __builtin_amdgcn_sched_barrier(0)

__device__ __forceinline__ void kt_decode(int j, int rank, int& p, int& s) {
  if (j < 8) { p = rank; s = j; }
  else { const int jj = j - 8; s = jj / 7; const int pi = jj - s * 7; p = (rank + 1 + pi) & 7; }
}
#define PUSH_STEP 4
#define PUSH_LAG 3
#define PUSH_T0 1
#define OUT_AUX 0
#define PF_ON 1
#define PF_W 0
#define PF_DW 7
#define AUX_W 2
#define AUX_A 0
#define EARLY_INV 0

__device__ __forceinline__ u64 ld_sys64(const char* p) { return __hip_atomic_load((const u64*)p, __ATOMIC_RELAXED, __HIP_MEMORY_SCOPE_SYSTEM); }
__device__ __forceinline__ void st_sys64(char* p, u64 v) { __hip_atomic_store((u64*)p, v, __ATOMIC_RELAXED, __HIP_MEMORY_SCOPE_SYSTEM); }
__device__ __forceinline__ unsigned ld_agent32(const char* p) { return __hip_atomic_load((const unsigned*)p, __ATOMIC_RELAXED, __HIP_MEMORY_SCOPE_AGENT); }
__device__ __forceinline__ void st_agent32(char* p, unsigned v) { __hip_atomic_store((unsigned*)p, v, __ATOMIC_RELAXED, __HIP_MEMORY_SCOPE_AGENT); }

__device__ __forceinline__ bool give_up(char* myws, unsigned call, u64 t0, unsigned& spins) {
  if ((++spins & 15u) != 0u) return false;
  const unsigned tm = __builtin_amdgcn_readfirstlane(ld_agent32(myws + WS_TMO));
  if (tm == call) return true;
  if (__builtin_amdgcn_s_memrealtime() - t0 > DEADLINE_TICKS) { st_agent32(myws + WS_TMO, call); return true; }
  return false;
}

__device__ __forceinline__ unsigned wave_umax(unsigned v) {
  v = max(v, (unsigned)__builtin_amdgcn_update_dpp(0, (int)v, 0x111, 0xf, 0xf, false));
  v = max(v, (unsigned)__builtin_amdgcn_update_dpp(0, (int)v, 0x112, 0xf, 0xf, false));
  v = max(v, (unsigned)__builtin_amdgcn_update_dpp(0, (int)v, 0x114, 0xf, 0xf, false));
  v = max(v, (unsigned)__builtin_amdgcn_update_dpp(0, (int)v, 0x118, 0xf, 0xf, false));
  v = max(v, (unsigned)__builtin_amdgcn_update_dpp(0, (int)v, 0x142, 0xa, 0xf, false));
  v = max(v, (unsigned)__builtin_amdgcn_update_dpp(0, (int)v, 0x143, 0xc, 0xf, false));
  return v;
}

#define STAMP(i) do { } while (0)

__global__ __launch_bounds__(512, 2) void a2a_gemm_q8(Params P) {
  extern __shared__ __attribute__((aligned(16))) char smem[];
  LAS char* lds = (LAS char*)smem;
  const int tid = threadIdx.x;
  const int wid = __builtin_amdgcn_readfirstlane(tid >> 6);
  const int lane = tid & 63;
  const int b = blockIdx.x;
  const int xcd = b & 7, idx = b >> 3;
  const int mt = idx & 1, nt = xcd * 16 + (idx >> 1);
  const int rank = P.rank;
  char* const myws = P.ws[rank];
  unsigned call = 0;

  if (wid >= 4) {
    if (wid < 6) {
      const int h = wid - 4;
      const int r8 = lane >> 3, c8 = lane & 7;
      const unsigned va0 = (unsigned)(r8 * 1024 + ((c8 ^ ((0 + (lane >> 4)) & 7)) * 16));
      const unsigned va1 = (unsigned)(r8 * 1024 + ((c8 ^ ((4 + (lane >> 4)) & 7)) * 16));
      const unsigned rowbase = (unsigned)((mt * 256 + h * 128) * 1024);
      const int g4 = lane >> 4, c16 = lane & 15;
      const unsigned vw0 = (unsigned)(g4 * 32768 + ((c16 ^ ((g4 << 2) | 0)) * 16));
      const unsigned vw1 = (unsigned)(g4 * 32768 + ((c16 ^ ((g4 << 2) | 2)) * 16));
      auto wrsrc = __builtin_amdgcn_make_buffer_rsrc((void*)(P.W + nt * 256), 0, (int)((size_t)KTOT * NTOT * 2 - (size_t)nt * 256), 0x00020000);
      const char* const xloc = P.x + (size_t)rank * CHUNK_BYTES;
      auto issueA = [&](int j) {
        int p, s; kt_decode(j, rank, p, s);
        const char* base = (j < 8) ? xloc : (myws + WS_STG + (size_t)p * CHUNK_BYTES);
        auto rsrc = __builtin_amdgcn_make_buffer_rsrc((void*)base, 0, (int)CHUNK_BYTES, 0x00020000);
        LAS char* dst = lds + LDS_A + (j % NSA) * A_STAGE + h * 16384;
        const unsigned so = rowbase + s * 128;
#pragma unroll
        for (int i = 0; i < 16; ++i)
          __builtin_amdgcn_raw_ptr_buffer_load_lds(rsrc, (LAS void*)(dst + i * 1024), 16, (i & 1) ? va1 : va0, so + i * 8192, 0, AUX_A);
      };
      auto issueW = [&](int j) {
        int p, s; kt_decode(j, rank, p, s);
        const unsigned so = (unsigned)(p * 512 + s * 64) * 32768u + (unsigned)h * (8u * 131072u);
        LAS char* dst = lds + LDS_B + (j % NSB) * B_STAGE + h * 8192;
#pragma unroll
        for (int i = 0; i < 8; ++i)
          __builtin_amdgcn_raw_ptr_buffer_load_lds(wrsrc, (LAS void*)(dst + i * 1024), 16, (i & 2) ? vw1 : vw0, so + i * 131072, 0, AUX_W);
      };
      issueA(0); issueW(0); issueW(1);
      WAIT_VM(8);
      BAR_ONLY();
      issueA(1); issueW(2);
      WAIT_VM(24); BAR_ONLY();
      issueA(2); issueW(3);
      for (int t = 0; t < NKT; ++t) {
        if (t + 3 < NKT) WAIT_VM(24); else if (t + 2 < NKT) WAIT_VM(16); else WAIT_VM(0);
        BAR_ONLY();
        if (t + 3 < NKT) issueA(t + 3);
        if (t + 4 < NKT) issueW(t + 4);
      }
    } else if (wid == 6) {
      const bool active = b < 7 * 32;
      const int pj = b % 7, pu = b / 7;
      const int dest = (rank + 1 + pj) & 7;
      u32x4 v[16];
      const char* src = P.x + (size_t)dest * CHUNK_BYTES + (size_t)((pu & 31) * 16 + (lane >> 3)) * 1024 + (lane & 7) * 16;
      v[0] = *(const u32x4*)(src); v[1] = *(const u32x4*)(src + 8192);
      const unsigned callv = ld_agent32(myws + WS_CC + b * 4);
      asm volatile("" ::: "memory");
#pragma unroll
      for (int i = 2; i < 16; ++i) v[i] = *(const u32x4*)(src + (i & 1) * 8192 + (i >> 1) * 128);
      asm volatile("" ::: "memory");
      const unsigned pcall = __builtin_amdgcn_readfirstlane(callv) + 1u;
      STAMP(0);
      auto drsrc = __builtin_amdgcn_make_buffer_rsrc((void*)(P.ws[dest] + WS_STG + (size_t)rank * CHUNK_BYTES), 0, (int)CHUNK_BYTES, 0x00020000);
      const unsigned voff = (unsigned)(((pu & 31) * 16 + (lane >> 3)) * 1024 + (lane & 7) * 16);
      char* fl = P.ws[dest] + WS_XF + ((size_t)((lane & (XF_REPS - 1)) * 2 + ((pu >> 4) & 1)) * NR + rank) * 128 + (pu & 15) * 4;
      auto push = [&](auto lo, auto hi) {
#pragma unroll
        for (int i = decltype(lo)::value; i < decltype(hi)::value; ++i)
          __builtin_amdgcn_raw_buffer_store_b128(v[i], drsrc, voff + (i & 1) * 8192 + (i >> 1) * 128, 0, 17  );
      };
      auto flag = [&](unsigned nslices, auto younger) {
        asm volatile("s_waitcnt vmcnt(%0)" :: "n"(decltype(younger)::value) : "memory");
        if (lane < XF_REPS) __hip_atomic_store((unsigned*)fl, pcall * 8u + nslices, __ATOMIC_RELAXED, __HIP_MEMORY_SCOPE_SYSTEM);
      };
      unsigned sink = 0;
      const char* const pfa = P.x + (size_t)rank * CHUNK_BYTES + (size_t)((mt * 256 + ((idx >> 1) & 15) * 16 + ((lane & 31) >> 1)) * 1024 + (lane & 1) * 64 + (lane >> 5) * 128);
      const char* const pfw = P.W + nt * 256 + (size_t)(mt * 32 + (lane >> 1)) * 32768 + (lane & 1) * 128;
      auto touchW = [&](int j) {
        int p, s; kt_decode(j, rank, p, s);
        const char* a = pfw + (size_t)(p * 512 + s * 64) * 32768u;
        asm volatile("global_load_dword %0, %1, off" : "+v"(sink) : "v"(a) : "memory");
      };
      int t = 0;
      auto bars_until = [&](int T) { for (; t < T; ++t) { if (PF_W) touchW(min(t + PF_DW, NKT - 1)); BAR_ONLY(); } };
      if (active) push(IC<0>{}, IC<2>{});
      STAMP(1);
      BAR_ONLY(); BAR_ONLY();
      asm volatile("global_load_dword %0, %1, off offset:384\n\tglobal_load_dword %0, %1, off offset:640\n\tglobal_load_dword %0, %1, off offset:768" : "+v"(sink) : "v"(pfa) : "memory");
      if (PF_W) { for (int j = 5; j < PF_DW; ++j) touchW(j); }
      if (active) {
        const int ph = pu & (PUSH_STEP - 1);
        bars_until(PUSH_T0); flag(1, IC<PF_ON * 3 + PF_W * (PF_DW - 5 + PUSH_T0)>{}); STAMP(2);
#define PUSH_SLICE(S) bars_until(PUSH_STEP * (S) - 2 + ph); push(IC<2 * (S)>{}, IC<2 * (S) + 2>{}); bars_until(PUSH_STEP * (S) - 2 + ph + PUSH_LAG); flag((S) + 1, IC<PF_W * PUSH_LAG>{});
        PUSH_SLICE(1) STAMP(3);
        PUSH_SLICE(2) PUSH_SLICE(3) PUSH_SLICE(4) PUSH_SLICE(5) PUSH_SLICE(6) PUSH_SLICE(7)
        STAMP(4);
#undef PUSH_SLICE
      }
      bars_until(NKT);
      asm volatile("s_waitcnt vmcnt(0)" :: "v"(sink) : "memory");
    } else {
      call = __builtin_amdgcn_readfirstlane(ld_agent32(myws + WS_CC + b * 4)) + 1u;
      const int pi = lane >> 3;
      const int src = (rank + 1 + pi) & 7;
      const char* fl = myws + WS_XF + ((size_t)(((idx >> 1) & (XF_REPS - 1)) * 2 + mt) * NR + src) * 128 + (lane & 7) * 8;
      auto flags_ok = [&](u64 v, unsigned need) { return pi == 7 || ((unsigned)v >= need && (unsigned)(v >> 32) >= need); };
      BAR_ONLY(); BAR_ONLY();
      u64 pre = 0;
      for (int t = 0; t < NKT; ++t) {
        const int j = t + 3 - 8;
        if (j >= 0 && j < 56 && j % 7 == 0) {
          const unsigned need = call * 8u + (unsigned)(j / 7) + 1u;
          if (!__all(flags_ok(pre, need))) {
            const u64 t0 = __builtin_amdgcn_s_memrealtime(); unsigned spins = 0;
            for (;;) {
              const u64 v = ld_sys64(fl);
              if (__all(flags_ok(v, need))) break;
              if (give_up(myws, call, t0, spins)) break;
              __builtin_amdgcn_s_sleep(1);
            }
          }
          if (j == 0) STAMP(5); if (j == 7) STAMP(6); if (j == 49) STAMP(7);
        }
        if (j + 1 >= 0 && j + 1 < 56 && (j + 1) % 7 == 0 && !__all(flags_ok(pre, call * 8u + (unsigned)((j + 1) / 7) + 1u))) pre = ld_sys64(fl);
        BAR_ONLY();
      }
    }
  } else {
    const int wm = wid >> 1, wn = wid & 1;
    const int l15 = lane & 15, g = lane >> 4, q = l15 >> 2, pp = lane & 3;
    unsigned baseA[2], baseB[4];
#pragma unroll
    for (int ks = 0; ks < 2; ++ks) baseA[ks] = (unsigned)((wm * 128 + l15) * 128 + (((4 * ks + g) ^ (l15 >> 1)) * 16)) + LDS_A;
    const int swz = (q << 2) | ((g & 1) << 1);
#pragma unroll
    for (int c = 0; c < 4; ++c) baseB[c] = (unsigned)((8 * g + q) * 256 + (((8 * wn + 2 * c + (pp >> 1)) ^ swz) * 16) + 8 * (pp & 1)) + LDS_B;

    f32x4 acc[8][4];
#pragma unroll
    for (int m = 0; m < 8; ++m)
#pragma unroll
      for (int c = 0; c < 4; ++c) acc[m][c] = (f32x4){0.f, 0.f, 0.f, 0.f};

    const unsigned callv = ld_agent32(myws + WS_CC + b * 4);
    __builtin_amdgcn_s_setprio(1);
    if (wid == 0) STAMP(10);
    BAR_ONLY();
    if (wid == 0) STAMP(11);
    {
      bf16x8 bx[4][2], by[4][2], a0[2], a1[2];
      auto rdB1 = [&](bf16x8& dst, unsigned sb, int c, int ks) {
        const s16x4 lo = __builtin_amdgcn_ds_read_tr16_b64_v4i16((LAS s16x4*)(lds + (baseB[c] + sb) + ks * 8192));
        const s16x4 hi = __builtin_amdgcn_ds_read_tr16_b64_v4i16((LAS s16x4*)(lds + (baseB[c] + sb) + ks * 8192 + 1024));
        dst = __builtin_shufflevector(lo, hi, 0, 1, 2, 3, 4, 5, 6, 7);
      };
      auto rdA1 = [&](bf16x8 (&dst)[2], unsigned sa, int m) {
        dst[0] = *(const LAS bf16x8*)(lds + (baseA[0] + sa) + m * 2048);
        dst[1] = *(const LAS bf16x8*)(lds + (baseA[1] + sa) + m * 2048);
      };
#pragma unroll
      for (int c = 0; c < 4; ++c) { rdB1(bx[c][0], 0, c, 0); rdB1(bx[c][1], 0, c, 1); }
      rdA1(a0, 0, 0); rdA1(a1, 0, 1);
      BAR_LGKM();
      unsigned sa = 0, sbn = B_STAGE;
      auto tile = [&](bf16x8 (&bc)[4][2], bf16x8 (&bn)[4][2]) {
        unsigned san = sa + A_STAGE; if (san == NSA * A_STAGE) san = 0;
        bf16x8 a[10][2];
        a[0][0] = a0[0]; a[0][1] = a0[1]; a[1][0] = a1[0]; a[1][1] = a1[1];
#pragma unroll
        for (int m = 0; m < 8; ++m) {
#pragma unroll
          for (int i = 0; i < 4; ++i) {
            const int c = (m & 1) ? 3 - i : i;
            acc[m][c] = __builtin_amdgcn_mfma_f32_16x16x32_bf16(bc[c][0], a[m][0], acc[m][c], 0, 0, 0);
            if (i == 0) { if (m + 2 < 8) rdA1(a[m + 2], sa, m + 2); else rdA1(a[m + 2], san, m - 6); }
            if (i == 1 && m < 4) rdB1(bn[m][0], sbn, m, 0);
            if (i == 2 && m < 4) rdB1(bn[m][1], sbn, m, 1);
            acc[m][c] = __builtin_amdgcn_mfma_f32_16x16x32_bf16(bc[c][1], a[m][1], acc[m][c], 0, 0, 0);
            if (i == 0 || ((i == 1 || i == 2) && m < 4)) {
              __builtin_amdgcn_sched_group_barrier(0x8, 1, 0); __builtin_amdgcn_sched_group_barrier(0x100, 1, 0);
              __builtin_amdgcn_sched_group_barrier(0x8, 1, 0); __builtin_amdgcn_sched_group_barrier(0x100, 1, 0);
            }
            SCHED_FENCE();
          }
          if (m == 5) { BAR_LGKM(); SCHED_FENCE(); }
        }
        a0[0] = a[8][0]; a0[1] = a[8][1]; a1[0] = a[9][0]; a1[1] = a[9][1];
        sa = san;
        sbn += B_STAGE; if (sbn == NSB * B_STAGE) sbn = 0;
      };
      for (int t = 0; t < NKT; t += 2) { tile(bx, by); tile(by, bx); }
    }
    __builtin_amdgcn_s_setprio(0);
    if (wid == 0) STAMP(12);
    call = __builtin_amdgcn_readfirstlane(callv) + 1u;
    float am0 = 0.f, am1 = 0.f;
#pragma unroll
    for (int m = 0; m < 8; ++m)
#pragma unroll
      for (int c = 0; c < 4; ++c) {
        am0 = fmaxf(fmaxf(am0, fabsf(acc[m][c][0])), fabsf(acc[m][c][1]));
        am1 = fmaxf(fmaxf(am1, fabsf(acc[m][c][2])), fabsf(acc[m][c][3]));
      }
    const unsigned wmax = wave_umax(__float_as_uint(fmaxf(am0, am1)));
    if (lane == 63) __hip_atomic_store((u64*)(myws + WS_LG + (b * 4 + wid) * 8), ((u64)call << 32) | wmax, __ATOMIC_RELAXED, __HIP_MEMORY_SCOPE_AGENT);
    if (wid == 0) STAMP(8);
#pragma unroll
    for (int m = 0; m < 8; ++m)
#pragma unroll
      for (int c = 0; c < 4; ++c) {
        const int row = wm * 128 + m * 16 + l15;
        *(LAS f32x4*)(lds + row * 512 + (((wn * 16 + c * 4 + g) ^ (l15 & 7)) * 16)) = acc[m][c];
      }
  }

  BAR_LGKM();
  f32x4 yv[4][4];
  {
    const int r3 = lane >> 3, s8 = lane & 7;
#pragma unroll
    for (int i = 0; i < 4; ++i)
#pragma unroll
      for (int j = 0; j < 4; ++j)
        yv[i][j] = *(const LAS f32x4*)(lds + (wid * 32 + i * 8 + r3) * 512 + (((s8 * 4 + j) ^ r3) * 16));
  }
  if (wid == 7) {
    if (b == 0) {
      const u64 t0 = __builtin_amdgcn_s_memrealtime(); unsigned spins = 0;
      unsigned mx;
      for (;;) {
        bool ok = true; mx = 0;
#pragma unroll
        for (int i = 0; i < 16; ++i) {
          const u64 v = __hip_atomic_load((const u64*)(myws + WS_LG + (i * 64 + lane) * 8), __ATOMIC_RELAXED, __HIP_MEMORY_SCOPE_AGENT);
          ok = ok && (unsigned)(v >> 32) >= call; mx = max(mx, (unsigned)v);
        }
        if (__all(ok)) break;
        if (give_up(myws, call, t0, spins)) break;
      }
      mx = wave_umax(mx);
      mx = __builtin_amdgcn_readlane(mx, 63);
      st_sys64(P.ws[lane >> 3] + WS_FG + (lane & 7) * 128 + rank * 8, ((u64)call << 32) | mx);
    }
    unsigned gmx;
    {
      const char* fg = myws + WS_FG + xcd * 128 + (lane & 7) * 8;
      const u64 t0 = __builtin_amdgcn_s_memrealtime(); unsigned spins = 0;
      for (;;) {
        const u64 v = ld_sys64(fg);
        gmx = (unsigned)v;
        if (__all((unsigned)(v >> 32) >= call)) break;
        if (give_up(myws, call, t0, spins)) break;
      }
      gmx = __builtin_amdgcn_readlane(wave_umax(gmx), 63);
    }
    const unsigned tm = __builtin_amdgcn_readfirstlane(ld_agent32(myws + WS_TMO));
    if (lane == 0) {
      LAS unsigned* o = (LAS unsigned*)(lds + LDS_MISC + 64);
      const float scale = __uint_as_float(gmx) / 127.0f;
      o[0] = __float_as_uint(1.0f / scale);
      o[1] = (tm == call) ? 1u : 0u;
      st_agent32(myws + WS_CC + b * 4, call);
    }
    STAMP(9);
  }
  BAR_LGKM();
  const float inv = *(const LAS float*)(lds + LDS_MISC + 64);
  const unsigned bad = *(const LAS unsigned*)(lds + LDS_MISC + 68);
  if (wid == 0) STAMP(13);
  {
    auto orsrc = __builtin_amdgcn_make_buffer_rsrc((void*)P.out, 0, MLOC * NTOT, 0x00020000);
    const unsigned ovoff = (unsigned)((mt * 256 + wid * 32 + (lane >> 3)) * NTOT + nt * 128 + (lane & 7) * 16);
#pragma unroll
    for (int i = 0; i < 4; ++i) {
      unsigned d[4];
#pragma unroll
      for (int j = 0; j < 4; ++j) {
        unsigned w = 0;
#pragma unroll
        for (int r = 0; r < 4; ++r) {
          float u = __builtin_fmaf(yv[i][j][r], inv, 128.0f);
          u = __builtin_amdgcn_fmed3f(u, 1.0f, 255.0f);
          w = __builtin_amdgcn_cvt_pk_u8_f32(u, r, w);
        }
        d[j] = bad ? 0x80808080u : (w ^ 0x80808080u);
      }
      const u32x4 v = {d[0], d[1], d[2], d[3]};
      __builtin_amdgcn_raw_buffer_store_b128(v, orsrc, ovoff + i * 8 * NTOT, 0, OUT_AUX);
    }
  }
}

}

extern "C" void kernel_launch(int rank, int nranks, ncclComm_t comm,
                              hipStream_t stream,
                              void* const* d_in, const int* in_sizes, int n_in,
                              void* d_out, int out_size,
                              const int* in_dtypes, int out_dtype,
                              void* const* d_out_all,
                              void* const* d_ws_all, size_t ws_bytes) {
  (void)comm; (void)in_dtypes; (void)out_dtype; (void)d_out_all;
  if (nranks != NR || n_in != 2 || (unsigned)rank >= (unsigned)NR || ws_bytes < WS_END + 131072 ||
      in_sizes[0] != KTOT * 512 || in_sizes[1] != KTOT * NTOT || out_size != MLOC * NTOT) {
    fprintf(stderr, "unexpected problem shape\n");
    return;
  }
  HIP_CHECK(hipFuncSetAttribute((const void*)a2a_gemm_q8, hipFuncAttributeMaxDynamicSharedMemorySize, LDS_BYTES));
  Params P;
  P.x = (const char*)d_in[0];
  P.W = (const char*)d_in[1];
  P.out = (signed char*)d_out;
  for (int p = 0; p < NR; ++p) P.ws[p] = (char*)d_ws_all[p];
  P.rank = rank;
  hipLaunchKernelGGL(a2a_gemm_q8, dim3(NWG), dim3(512), LDS_BYTES, stream, P);
  HIP_CHECK(hipGetLastError());
}
```

```cpp
#include <hip/hip_runtime.h>
#include <rccl/rccl.h>
#include <cstdio>

#define HIP_CHECK(expr) do { hipError_t _e = (expr); if (_e != hipSuccess) { \
    fprintf(stderr, "HIP error %s at %s:%d\n", hipGetErrorString(_e), __FILE__, __LINE__); \
    return; } } while (0)

#define LAS __attribute__((address_space(3)))
typedef short bf16x8 __attribute__((ext_vector_type(8)));
typedef short s16x4 __attribute__((ext_vector_type(4)));
typedef float f32x4 __attribute__((ext_vector_type(4)));
typedef unsigned u32x4 __attribute__((ext_vector_type(4)));
typedef unsigned long long u64;

namespace {

constexpr int NR = 8;
constexpr int MLOC = 512, KTOT = 4096, NTOT = 16384;
constexpr int BM = 256, BN = 128, BK = 64, NKT = KTOT / BK;
constexpr int NWG = 256;
constexpr int A_STAGE = BM * BK * 2;
constexpr int B_STAGE = BK * BN * 2;
#define NSB_ 3
constexpr int NSA = 3, NSB = NSB_;
constexpr int LDS_A = 0, LDS_B = NSA * A_STAGE;
constexpr int LDS_BYTES = LDS_B + NSB * B_STAGE;
constexpr int LDS_MISC = 131072;
static_assert(LDS_BYTES <= 163840, "LDS");

constexpr size_t CHUNK_BYTES = (size_t)MLOC * 512 * 2;
constexpr size_t WS_STG = 0;
constexpr size_t WS_XF = WS_STG + NR * CHUNK_BYTES;
constexpr int XF_REPS = 4;
constexpr size_t WS_XF_BYTES = (size_t)XF_REPS * 2 * NR * 128;
constexpr size_t WS_LG = WS_XF + WS_XF_BYTES;
constexpr size_t WS_FG = WS_LG + NWG * 4 * 8;
#define FG_REPS_ 8
constexpr int FG_REPS = FG_REPS_;
constexpr size_t WS_CC = WS_FG + FG_REPS * 128;
constexpr size_t WS_TMO = WS_CC + NWG * 4;
constexpr size_t WS_END = WS_TMO + 128;

constexpr u64 DEADLINE_TICKS = 50000000ull;

struct Params {
  const char* x;
  const char* W;
  signed char* out;
  char* ws[NR];
  int rank;
};

template <int N> struct IC { static constexpr int value = N; };

#define BAR_LGKM() asm volatile("s_waitcnt lgkmcnt(0)\n\ts_barrier" ::: "memory")
#define BAR_ONLY() asm volatile("s_barrier" ::: "memory")
#define WAIT_VM(n) asm volatile("s_waitcnt vmcnt(" #n ")" ::: "memory")
#define SCHED_FENCE() __builtin_amdgcn_sched_barrier(0)

__device__ __forceinline__ void kt_decode(int j, int rank, int& p, int& s) {
  if (j < 8) { p = rank; s = j; }
  else { const int jj = j - 8; s = jj / 7; const int pi = jj - s * 7; p = (rank + 1 + pi) & 7; }
}
#define PUSH_STEP 4
#define PUSH_LAG 3
#define PUSH_T0 1
#define OUT_AUX 16
#define PF_ON 1
#define PF_W 0
#define PF_DW 7
#define AUX_W 0
#define AUX_A 0
#define EARLY_INV 0

__device__ __forceinline__ u64 ld_sys64(const char* p) { return __hip_atomic_load((const u64*)p, __ATOMIC_RELAXED, __HIP_MEMORY_SCOPE_SYSTEM); }
__device__ __forceinline__ void st_sys64(char* p, u64 v) { __hip_atomic_store((u64*)p, v, __ATOMIC_RELAXED, __HIP_MEMORY_SCOPE_SYSTEM); }
__device__ __forceinline__ unsigned ld_agent32(const char* p) { return __hip_atomic_load((const unsigned*)p, __ATOMIC_RELAXED, __HIP_MEMORY_SCOPE_AGENT); }
__device__ __forceinline__ void st_agent32(char* p, unsigned v) { __hip_atomic_store((unsigned*)p, v, __ATOMIC_RELAXED, __HIP_MEMORY_SCOPE_AGENT); }

__device__ __forceinline__ bool give_up(char* myws, unsigned call, u64 t0, unsigned& spins) {
  if ((++spins & 15u) != 0u) return false;
  const unsigned tm = __builtin_amdgcn_readfirstlane(ld_agent32(myws + WS_TMO));
  if (tm == call) return true;
  if (__builtin_amdgcn_s_memrealtime() - t0 > DEADLINE_TICKS) {
    st_agent32(myws + WS_TMO, call);
    asm volatile("s_waitcnt vmcnt(0)" ::: "memory");
    const u64 t1 = __builtin_amdgcn_s_memrealtime();
    while (__builtin_amdgcn_s_memrealtime() - t1 < 1000ull) __builtin_amdgcn_s_sleep(8);
    return true;
  }
  return false;
}

__device__ __forceinline__ unsigned wave_umax(unsigned v) {
  v = max(v, (unsigned)__builtin_amdgcn_update_dpp(0, (int)v, 0x111, 0xf, 0xf, false));
  v = max(v, (unsigned)__builtin_amdgcn_update_dpp(0, (int)v, 0x112, 0xf, 0xf, false));
  v = max(v, (unsigned)__builtin_amdgcn_update_dpp(0, (int)v, 0x114, 0xf, 0xf, false));
  v = max(v, (unsigned)__builtin_amdgcn_update_dpp(0, (int)v, 0x118, 0xf, 0xf, false));
  v = max(v, (unsigned)__builtin_amdgcn_update_dpp(0, (int)v, 0x142, 0xa, 0xf, false));
  v = max(v, (unsigned)__builtin_amdgcn_update_dpp(0, (int)v, 0x143, 0xc, 0xf, false));
  return v;
}

#define STAMP(i) do { } while (0)

__global__ __launch_bounds__(512, 2) void a2a_gemm_q8(Params P) {
  extern __shared__ __attribute__((aligned(16))) char smem[];
  LAS char* lds = (LAS char*)smem;
  const int tid = threadIdx.x;
  const int wid = __builtin_amdgcn_readfirstlane(tid >> 6);
  const int lane = tid & 63;
  const int b = blockIdx.x;
  const int xcd = b & 7, idx = b >> 3;
  const int mt = idx & 1, nt = xcd * 16 + (idx >> 1);
  const int rank = P.rank;
  char* const myws = P.ws[rank];
  unsigned call = 0;
  bool timed_out = false;

  if (wid >= 4) {
    if (wid < 6) {
      const int h = wid - 4;
      const int r8 = lane >> 3, c8 = lane & 7;
      const unsigned va0 = (unsigned)(r8 * 1024 + ((c8 ^ ((0 + (lane >> 4)) & 7)) * 16));
      const unsigned va1 = (unsigned)(r8 * 1024 + ((c8 ^ ((4 + (lane >> 4)) & 7)) * 16));
      const unsigned rowbase = (unsigned)((mt * 256 + h * 128) * 1024);
      const int g4 = lane >> 4, c16 = lane & 15;
      const unsigned vw0 = (unsigned)(g4 * 32768 + ((c16 ^ ((g4 << 2) | 0)) * 16));
      const unsigned vw1 = (unsigned)(g4 * 32768 + ((c16 ^ ((g4 << 2) | 2)) * 16));
      auto wrsrc = __builtin_amdgcn_make_buffer_rsrc((void*)(P.W + nt * 256), 0, (int)((size_t)KTOT * NTOT * 2 - (size_t)nt * 256), 0x00020000);
      const char* const xloc = P.x + (size_t)rank * CHUNK_BYTES;
      auto issueA = [&](int j) {
        int p, s; kt_decode(j, rank, p, s);
        const char* base = (j < 8) ? xloc : (myws + WS_STG + (size_t)p * CHUNK_BYTES);
        auto rsrc = __builtin_amdgcn_make_buffer_rsrc((void*)base, 0, (int)CHUNK_BYTES, 0x00020000);
        LAS char* dst = lds + LDS_A + (j % NSA) * A_STAGE + h * 16384;
        const unsigned so = rowbase + s * 128;
#pragma unroll
        for (int i = 0; i < 16; ++i)
          __builtin_amdgcn_raw_ptr_buffer_load_lds(rsrc, (LAS void*)(dst + i * 1024), 16, (i & 1) ? va1 : va0, so + i * 8192, 0, AUX_A);
      };
      auto issueW = [&](int j) {
        int p, s; kt_decode(j, rank, p, s);
        const unsigned so = (unsigned)(p * 512 + s * 64) * 32768u + (unsigned)h * (8u * 131072u);
        LAS char* dst = lds + LDS_B + (j % NSB) * B_STAGE + h * 8192;
#pragma unroll
        for (int i = 0; i < 8; ++i)
          __builtin_amdgcn_raw_ptr_buffer_load_lds(wrsrc, (LAS void*)(dst + i * 1024), 16, (i & 2) ? vw1 : vw0, so + i * 131072, 0, AUX_W);
      };
      issueA(0); issueW(0); issueW(1);
      WAIT_VM(8);
      BAR_ONLY();
      issueA(1); issueW(2);
      WAIT_VM(24); BAR_ONLY();
      issueA(2); issueW(3);
      for (int t = 0; t < NKT; ++t) {
        if (t + 3 < NKT) WAIT_VM(24); else if (t + 2 < NKT) WAIT_VM(16); else WAIT_VM(0);
        BAR_ONLY();
        if (t + 3 < NKT) issueA(t + 3);
        if (t + 4 < NKT) issueW(t + 4);
      }
    } else if (wid == 6) {
      const bool active = b < 7 * 32;
      const int pj = b % 7, pu = b / 7;
      const int dest = (rank + 1 + pj) & 7;
      u32x4 v[16];
      const char* src = P.x + (size_t)dest * CHUNK_BYTES + (size_t)((pu & 31) * 16 + (lane >> 3)) * 1024 + (lane & 7) * 16;
      v[0] = *(const u32x4*)(src); v[1] = *(const u32x4*)(src + 8192);
      const unsigned callv = ld_agent32(myws + WS_CC + b * 4);
      asm volatile("" ::: "memory");
#pragma unroll
      for (int i = 2; i < 16; ++i) v[i] = *(const u32x4*)(src + (i & 1) * 8192 + (i >> 1) * 128);
      asm volatile("" ::: "memory");
      const unsigned pcall = __builtin_amdgcn_readfirstlane(callv) + 1u;
      STAMP(0);
      auto drsrc = __builtin_amdgcn_make_buffer_rsrc((void*)(P.ws[dest] + WS_STG + (size_t)rank * CHUNK_BYTES), 0, (int)CHUNK_BYTES, 0x00020000);
      const unsigned voff = (unsigned)(((pu & 31) * 16 + (lane >> 3)) * 1024 + (lane & 7) * 16);
      char* fl = P.ws[dest] + WS_XF + ((size_t)((lane & (XF_REPS - 1)) * 2 + ((pu >> 4) & 1)) * NR + rank) * 128 + (pu & 15) * 4;
      auto push = [&](auto lo, auto hi) {
#pragma unroll
        for (int i = decltype(lo)::value; i < decltype(hi)::value; ++i)
          __builtin_amdgcn_raw_buffer_store_b128(v[i], drsrc, voff + (i & 1) * 8192 + (i >> 1) * 128, 0, 17  );
      };
      auto flag = [&](unsigned nslices, auto younger) {
        asm volatile("s_waitcnt vmcnt(%0)" :: "n"(decltype(younger)::value) : "memory");
        if (lane < XF_REPS) __hip_atomic_store((unsigned*)fl, pcall * 8u + nslices, __ATOMIC_RELAXED, __HIP_MEMORY_SCOPE_SYSTEM);
      };
      unsigned sink = 0;
      const char* const pfa = P.x + (size_t)rank * CHUNK_BYTES + (size_t)((mt * 256 + ((idx >> 1) & 15) * 16 + ((lane & 31) >> 1)) * 1024 + (lane & 1) * 64 + (lane >> 5) * 128);
      const char* const pfw = P.W + nt * 256 + (size_t)(mt * 32 + (lane >> 1)) * 32768 + (lane & 1) * 128;
      auto touchW = [&](int j) {
        int p, s; kt_decode(j, rank, p, s);
        const char* a = pfw + (size_t)(p * 512 + s * 64) * 32768u;
        asm volatile("global_load_dword %0, %1, off" : "+v"(sink) : "v"(a) : "memory");
      };
      int t = 0;
      auto bars_until = [&](int T) { for (; t < T; ++t) { if (PF_W) touchW(min(t + PF_DW, NKT - 1)); BAR_ONLY(); } };
      if (active) push(IC<0>{}, IC<2>{});
      STAMP(1);
      BAR_ONLY(); BAR_ONLY();
      asm volatile("global_load_dword %0, %1, off offset:384\n\tglobal_load_dword %0, %1, off offset:640\n\tglobal_load_dword %0, %1, off offset:768" : "+v"(sink) : "v"(pfa) : "memory");
      if (PF_W) { for (int j = 5; j < PF_DW; ++j) touchW(j); }
      if (active) {
        const int ph = pu & (PUSH_STEP - 1);
        bars_until(PUSH_T0); flag(1, IC<PF_ON * 3 + PF_W * (PF_DW - 5 + PUSH_T0)>{}); STAMP(2);
#define PUSH_SLICE(S) bars_until(PUSH_STEP * (S) - 2 + ph); push(IC<2 * (S)>{}, IC<2 * (S) + 2>{}); bars_until(PUSH_STEP * (S) - 2 + ph + PUSH_LAG); flag((S) + 1, IC<PF_W * PUSH_LAG>{});
        PUSH_SLICE(1) STAMP(3);
        PUSH_SLICE(2) PUSH_SLICE(3) PUSH_SLICE(4) PUSH_SLICE(5) PUSH_SLICE(6) PUSH_SLICE(7)
        STAMP(4);
#undef PUSH_SLICE
      }
      bars_until(NKT);
      asm volatile("s_waitcnt vmcnt(0)" :: "v"(sink) : "memory");
    } else {
      call = __builtin_amdgcn_readfirstlane(ld_agent32(myws + WS_CC + b * 4)) + 1u;
      const int pi = lane >> 3;
      const int src = (rank + 1 + pi) & 7;
      const char* fl = myws + WS_XF + ((size_t)(((idx >> 1) & (XF_REPS - 1)) * 2 + mt) * NR + src) * 128 + (lane & 7) * 8;
      auto flags_ok = [&](u64 v, unsigned need) { return pi == 7 || ((unsigned)v >= need && (unsigned)(v >> 32) >= need); };
      BAR_ONLY(); BAR_ONLY();
      u64 pre = 0;
      for (int t = 0; t < NKT; ++t) {
        const int j = t + 3 - 8;
        if (j >= 0 && j < 56 && j % 7 == 0) {
          const unsigned need = call * 8u + (unsigned)(j / 7) + 1u;
          if (!__all(flags_ok(pre, need))) {
            const u64 t0 = __builtin_amdgcn_s_memrealtime(); unsigned spins = 0;
            for (;;) {
              const u64 v = ld_sys64(fl);
              if (__all(flags_ok(v, need))) break;
              if (give_up(myws, call, t0, spins)) { timed_out = true; break; }
              __builtin_amdgcn_s_sleep(1);
            }
          }
          if (j == 0) STAMP(5); if (j == 7) STAMP(6); if (j == 49) STAMP(7);
        }
        if (j + 1 >= 0 && j + 1 < 56 && (j + 1) % 7 == 0 && !__all(flags_ok(pre, call * 8u + (unsigned)((j + 1) / 7) + 1u))) pre = ld_sys64(fl);
        BAR_ONLY();
      }
    }
  } else {
    const int wm = wid >> 1, wn = wid & 1;
    const int l15 = lane & 15, g = lane >> 4, q = l15 >> 2, pp = lane & 3;
    unsigned baseA[2], baseB[4];
#pragma unroll
    for (int ks = 0; ks < 2; ++ks) baseA[ks] = (unsigned)((wm * 128 + l15) * 128 + (((4 * ks + g) ^ (l15 >> 1)) * 16)) + LDS_A;
    const int swz = (q << 2) | ((g & 1) << 1);
#pragma unroll
    for (int c = 0; c < 4; ++c) baseB[c] = (unsigned)((8 * g + q) * 256 + (((8 * wn + 2 * c + (pp >> 1)) ^ swz) * 16) + 8 * (pp & 1)) + LDS_B;

    f32x4 acc[8][4];
#pragma unroll
    for (int m = 0; m < 8; ++m)
#pragma unroll
      for (int c = 0; c < 4; ++c) acc[m][c] = (f32x4){0.f, 0.f, 0.f, 0.f};

    const unsigned callv = ld_agent32(myws + WS_CC + b * 4);
    __builtin_amdgcn_s_setprio(1);
    if (wid == 0) STAMP(10);
    BAR_ONLY();
    if (wid == 0) STAMP(11);
    {
      bf16x8 bx[4][2], by[4][2], a0[2], a1[2];
      auto rdB1 = [&](bf16x8& dst, unsigned sb, int c, int ks) {
        const s16x4 lo = __builtin_amdgcn_ds_read_tr16_b64_v4i16((LAS s16x4*)(lds + (baseB[c] + sb) + ks * 8192));
        const s16x4 hi = __builtin_amdgcn_ds_read_tr16_b64_v4i16((LAS s16x4*)(lds + (baseB[c] + sb) + ks * 8192 + 1024));
        dst = __builtin_shufflevector(lo, hi, 0, 1, 2, 3, 4, 5, 6, 7);
      };
      auto rdA1 = [&](bf16x8 (&dst)[2], unsigned sa, int m) {
        dst[0] = *(const LAS bf16x8*)(lds + (baseA[0] + sa) + m * 2048);
        dst[1] = *(const LAS bf16x8*)(lds + (baseA[1] + sa) + m * 2048);
      };
#pragma unroll
      for (int c = 0; c < 4; ++c) { rdB1(bx[c][0], 0, c, 0); rdB1(bx[c][1], 0, c, 1); }
      rdA1(a0, 0, 0); rdA1(a1, 0, 1);
      BAR_LGKM();
      unsigned sa = 0, sbn = B_STAGE;
      auto tile = [&](bf16x8 (&bc)[4][2], bf16x8 (&bn)[4][2]) {
        unsigned san = sa + A_STAGE; if (san == NSA * A_STAGE) san = 0;
        bf16x8 a[10][2];
        a[0][0] = a0[0]; a[0][1] = a0[1]; a[1][0] = a1[0]; a[1][1] = a1[1];
#pragma unroll
        for (int m = 0; m < 8; ++m) {
#pragma unroll
          for (int i = 0; i < 4; ++i) {
            const int c = (m & 1) ? 3 - i : i;
            acc[m][c] = __builtin_amdgcn_mfma_f32_16x16x32_bf16(bc[c][0], a[m][0], acc[m][c], 0, 0, 0);
            if (i == 0) { if (m + 2 < 8) rdA1(a[m + 2], sa, m + 2); else rdA1(a[m + 2], san, m - 6); }
            if (i == 1 && m < 4) rdB1(bn[m][0], sbn, m, 0);
            if (i == 2 && m < 4) rdB1(bn[m][1], sbn, m, 1);
            acc[m][c] = __builtin_amdgcn_mfma_f32_16x16x32_bf16(bc[c][1], a[m][1], acc[m][c], 0, 0, 0);
            if (i == 0 || ((i == 1 || i == 2) && m < 4)) {
              __builtin_amdgcn_sched_group_barrier(0x8, 1, 0); __builtin_amdgcn_sched_group_barrier(0x100, 1, 0);
              __builtin_amdgcn_sched_group_barrier(0x8, 1, 0); __builtin_amdgcn_sched_group_barrier(0x100, 1, 0);
            }
            SCHED_FENCE();
          }
          if (m == 5) { BAR_LGKM(); SCHED_FENCE(); }
        }
        a0[0] = a[8][0]; a0[1] = a[8][1]; a1[0] = a[9][0]; a1[1] = a[9][1];
        sa = san;
        sbn += B_STAGE; if (sbn == NSB * B_STAGE) sbn = 0;
      };
      for (int t = 0; t < NKT; t += 2) { tile(bx, by); tile(by, bx); }
    }
    __builtin_amdgcn_s_setprio(0);
    if (wid == 0) STAMP(12);
    call = __builtin_amdgcn_readfirstlane(callv) + 1u;
    float am0 = 0.f, am1 = 0.f;
#pragma unroll
    for (int m = 0; m < 8; ++m)
#pragma unroll
      for (int c = 0; c < 4; ++c) {
        am0 = fmaxf(fmaxf(am0, fabsf(acc[m][c][0])), fabsf(acc[m][c][1]));
        am1 = fmaxf(fmaxf(am1, fabsf(acc[m][c][2])), fabsf(acc[m][c][3]));
      }
    const unsigned wmax = wave_umax(__float_as_uint(fmaxf(am0, am1)));
    if (lane == 63) __hip_atomic_store((u64*)(myws + WS_LG + (b * 4 + wid) * 8), ((u64)call << 32) | wmax, __ATOMIC_RELAXED, __HIP_MEMORY_SCOPE_AGENT);
    if (wid == 0) STAMP(8);
#pragma unroll
    for (int m = 0; m < 8; ++m)
#pragma unroll
      for (int c = 0; c < 4; ++c) {
        const int row = wm * 128 + m * 16 + l15;
        *(LAS f32x4*)(lds + row * 512 + (((wn * 16 + c * 4 + g) ^ (l15 & 7)) * 16)) = acc[m][c];
      }
  }

  BAR_LGKM();
  f32x4 yv[4][4];
  {
    const int r3 = lane >> 3, s8 = lane & 7;
#pragma unroll
    for (int i = 0; i < 4; ++i)
#pragma unroll
      for (int j = 0; j < 4; ++j)
        yv[i][j] = *(const LAS f32x4*)(lds + (wid * 32 + i * 8 + r3) * 512 + (((s8 * 4 + j) ^ r3) * 16));
  }
  if (wid == 7) {
    if (b == 0) {
      auto lgr = __builtin_amdgcn_make_buffer_rsrc((void*)(myws + WS_LG), 0, NWG * 4 * 8, 0x00020000);
      const u64 t0 = __builtin_amdgcn_s_memrealtime(); unsigned spins = 0;
      unsigned mx = 0;
      auto sweep = [&](u32x4 (&v)[8]) {
#pragma unroll
        for (int i = 0; i < 8; ++i) v[i] = __builtin_amdgcn_raw_buffer_load_b128(lgr, lane * 16 + i * 1024, 0, 16  );
        asm volatile("" ::: "memory");
      };
      auto check = [&](const u32x4 (&v)[8]) {
        bool ok = true; mx = 0;
#pragma unroll
        for (int i = 0; i < 8; ++i) { ok = ok && v[i][1] >= call && v[i][3] >= call; mx = max(mx, max(v[i][0], v[i][2])); }
        return __all(ok) != 0;
      };
      u32x4 va[8], vb[8];
      sweep(va);
      for (;;) {
        sweep(vb); if (check(va)) break;
        sweep(va); if (check(vb)) break;
        if (give_up(myws, call, t0, spins)) { timed_out = true; break; }
      }
      mx = __builtin_amdgcn_readlane(wave_umax(mx), 63);
      const u64 gr = ((u64)call << 32) | mx;
#pragma unroll
      for (int i = 0; i < FG_REPS / 8; ++i) st_sys64(P.ws[lane >> 3] + WS_FG + (i * 8 + (lane & 7)) * 128 + rank * 8, gr);
    }
    unsigned gmx, tm;
    {
      const char* fg = myws + WS_FG + (b & (FG_REPS - 1)) * 128 + (lane & 7) * 8;
      const u64 t0 = __builtin_amdgcn_s_memrealtime(); unsigned spins = 0;
      u64 v0 = ld_sys64(fg); unsigned m0 = ld_agent32(myws + WS_TMO);
      __builtin_amdgcn_s_sleep(3);
      u64 v1 = ld_sys64(fg); unsigned m1 = ld_agent32(myws + WS_TMO);
      __builtin_amdgcn_s_sleep(3);
      u64 v2 = ld_sys64(fg); unsigned m2 = ld_agent32(myws + WS_TMO);
      for (;;) {
        if (__all((unsigned)(v0 >> 32) >= call)) { gmx = (unsigned)v0; tm = m0; break; }
        v0 = ld_sys64(fg); m0 = ld_agent32(myws + WS_TMO);
        if (__all((unsigned)(v1 >> 32) >= call)) { gmx = (unsigned)v1; tm = m1; break; }
        v1 = ld_sys64(fg); m1 = ld_agent32(myws + WS_TMO);
        if (__all((unsigned)(v2 >> 32) >= call)) { gmx = (unsigned)v2; tm = m2; break; }
        v2 = ld_sys64(fg); m2 = ld_agent32(myws + WS_TMO);
        if (give_up(myws, call, t0, spins)) { gmx = 0; tm = call; break; }
      }
      gmx = __builtin_amdgcn_readlane(wave_umax(gmx), 63);
    }
    if (lane == 0) {
      LAS unsigned* o = (LAS unsigned*)(lds + LDS_MISC + 64);
      const float scale = __uint_as_float(gmx) / 127.0f;
      o[0] = __float_as_uint(1.0f / scale);
      o[1] = (tm == call || timed_out) ? 1u : 0u;
      st_agent32(myws + WS_CC + b * 4, call);
    }
    STAMP(9);
  }
  BAR_LGKM();
  const float inv = *(const LAS float*)(lds + LDS_MISC + 64);
  const unsigned bad = *(const LAS unsigned*)(lds + LDS_MISC + 68);
  if (wid == 0) STAMP(13);
  {
    auto orsrc = __builtin_amdgcn_make_buffer_rsrc((void*)P.out, 0, MLOC * NTOT, 0x00020000);
    const unsigned ovoff = (unsigned)((mt * 256 + wid * 32 + (lane >> 3)) * NTOT + nt * 128 + (lane & 7) * 16);
#pragma unroll
    for (int i = 0; i < 4; ++i) {
      unsigned d[4];
#pragma unroll
      for (int j = 0; j < 4; ++j) {
        unsigned w = 0;
#pragma unroll
        for (int r = 0; r < 4; ++r) {
          float u = __builtin_fmaf(yv[i][j][r], inv, 128.0f);
          u = __builtin_amdgcn_fmed3f(u, 1.0f, 255.0f);
          w = __builtin_amdgcn_cvt_pk_u8_f32(u, r, w);
        }
        d[j] = bad ? 0x80808080u : (w ^ 0x80808080u);
      }
      const u32x4 v = {d[0], d[1], d[2], d[3]};
      __builtin_amdgcn_raw_buffer_store_b128(v, orsrc, ovoff + i * 8 * NTOT, 0, OUT_AUX);
    }
  }
}

}

extern "C" void kernel_launch(int rank, int nranks, ncclComm_t comm,
                              hipStream_t stream,
                              void* const* d_in, const int* in_sizes, int n_in,
                              void* d_out, int out_size,
                              const int* in_dtypes, int out_dtype,
                              void* const* d_out_all,
                              void* const* d_ws_all, size_t ws_bytes) {
  (void)comm; (void)in_dtypes; (void)out_dtype; (void)d_out_all;
  if (nranks != NR || n_in != 2 || (unsigned)rank >= (unsigned)NR || ws_bytes < WS_END + 131072 ||
      in_sizes[0] != KTOT * 512 || in_sizes[1] != KTOT * NTOT || out_size != MLOC * NTOT) {
    fprintf(stderr, "unexpected problem shape\n");
    return;
  }
  HIP_CHECK(hipFuncSetAttribute((const void*)a2a_gemm_q8, hipFuncAttributeMaxDynamicSharedMemorySize, LDS_BYTES));
  Params P;
  P.x = (const char*)d_in[0];
  P.W = (const char*)d_in[1];
  P.out = (signed char*)d_out;
  for (int p = 0; p < NR; ++p) P.ws[p] = (char*)d_ws_all[p];
  P.rank = rank;
  hipLaunchKernelGGL(a2a_gemm_q8, dim3(NWG), dim3(512), LDS_BYTES, stream, P);
  HIP_CHECK(hipGetLastError());
}
```

```cpp
#include <hip/hip_runtime.h>
#include <rccl/rccl.h>
#include <cstdio>

#define HIP_CHECK(expr) do { hipError_t _e = (expr); if (_e != hipSuccess) { \
    fprintf(stderr, "HIP error %s at %s:%d\n", hipGetErrorString(_e), __FILE__, __LINE__); \
    return; } } while (0)

#define LAS __attribute__((address_space(3)))
typedef short bf16x8 __attribute__((ext_vector_type(8)));
typedef short s16x4 __attribute__((ext_vector_type(4)));
typedef float f32x4 __attribute__((ext_vector_type(4)));
typedef unsigned u32x4 __attribute__((ext_vector_type(4)));
typedef unsigned long long u64;

namespace {

constexpr int NR = 8;
constexpr int MLOC = 512, KTOT = 4096, NTOT = 16384;
constexpr int BM = 256, BN = 128, BK = 64, NKT = KTOT / BK;
constexpr int NWG = 256;
constexpr int A_STAGE = BM * BK * 2;
constexpr int B_STAGE = BK * BN * 2;
constexpr int NSA = 3, NSB = 3;
constexpr int LDS_A = 0, LDS_B = NSA * A_STAGE;
constexpr int LDS_BYTES = LDS_B + NSB * B_STAGE;
constexpr int LDS_MISC = 131072;
static_assert(LDS_BYTES <= 163840, "LDS");

constexpr size_t CHUNK_BYTES = (size_t)MLOC * 512 * 2;
constexpr size_t WS_STG = 0;
constexpr size_t WS_XF = WS_STG + NR * CHUNK_BYTES;
constexpr int XF_REPS = 4;
constexpr size_t WS_XF_BYTES = (size_t)XF_REPS * 2 * NR * 128;
constexpr size_t WS_LG = WS_XF + WS_XF_BYTES;
constexpr size_t WS_FG = WS_LG + NWG * 4 * 8;
constexpr int FG_REPS = 8;
constexpr size_t WS_CC = WS_FG + FG_REPS * 128;
constexpr size_t WS_TMO = WS_CC + NWG * 4;
constexpr size_t WS_END = WS_TMO + 128;

constexpr u64 DEADLINE_TICKS = 50000000ull;

struct Params {
  const char* x;
  const char* W;
  signed char* out;
  char* ws[NR];
  int rank;
};

template <int N> struct IC { static constexpr int value = N; };

#define BAR_LGKM() asm volatile("s_waitcnt lgkmcnt(0)\n\ts_barrier" ::: "memory")
#define BAR_ONLY() asm volatile("s_barrier" ::: "memory")
#define WAIT_VM(n) asm volatile("s_waitcnt vmcnt(" #n ")" ::: "memory")
#define SCHED_FENCE() __builtin_amdgcn_sched_barrier(0)

__device__ __forceinline__ void kt_decode(int j, int rank, int& p, int& s) {
  if (j < 8) { p = rank; s = j; }
  else { const int jj = j - 8; s = jj / 7; const int pi = jj - s * 7; p = (rank + 1 + pi) & 7; }
}
constexpr int PUSH_STEP = 4;
constexpr int PUSH_LAG = 3;
constexpr int PUSH_T0 = 1;
constexpr int OUT_AUX = 16;

__device__ __forceinline__ u64 ld_sys64(const char* p) { return __hip_atomic_load((const u64*)p, __ATOMIC_RELAXED, __HIP_MEMORY_SCOPE_SYSTEM); }
__device__ __forceinline__ void st_sys64(char* p, u64 v) { __hip_atomic_store((u64*)p, v, __ATOMIC_RELAXED, __HIP_MEMORY_SCOPE_SYSTEM); }
__device__ __forceinline__ unsigned ld_agent32(const char* p) { return __hip_atomic_load((const unsigned*)p, __ATOMIC_RELAXED, __HIP_MEMORY_SCOPE_AGENT); }
__device__ __forceinline__ void st_agent32(char* p, unsigned v) { __hip_atomic_store((unsigned*)p, v, __ATOMIC_RELAXED, __HIP_MEMORY_SCOPE_AGENT); }

__device__ __forceinline__ bool give_up(char* myws, unsigned call, u64 t0, unsigned& spins) {
  if ((++spins & 15u) != 0u) return false;
  const unsigned tm = __builtin_amdgcn_readfirstlane(ld_agent32(myws + WS_TMO));
  if (tm == call) return true;
  if (__builtin_amdgcn_s_memrealtime() - t0 > DEADLINE_TICKS) {
    st_agent32(myws + WS_TMO, call);
    asm volatile("s_waitcnt vmcnt(0)" ::: "memory");
    const u64 t1 = __builtin_amdgcn_s_memrealtime();
    while (__builtin_amdgcn_s_memrealtime() - t1 < 1000ull) __builtin_amdgcn_s_sleep(8);
    return true;
  }
  return false;
}

__device__ __forceinline__ unsigned wave_umax(unsigned v) {
  v = max(v, (unsigned)__builtin_amdgcn_update_dpp(0, (int)v, 0x111, 0xf, 0xf, false));
  v = max(v, (unsigned)__builtin_amdgcn_update_dpp(0, (int)v, 0x112, 0xf, 0xf, false));
  v = max(v, (unsigned)__builtin_amdgcn_update_dpp(0, (int)v, 0x114, 0xf, 0xf, false));
  v = max(v, (unsigned)__builtin_amdgcn_update_dpp(0, (int)v, 0x118, 0xf, 0xf, false));
  v = max(v, (unsigned)__builtin_amdgcn_update_dpp(0, (int)v, 0x142, 0xa, 0xf, false));
  v = max(v, (unsigned)__builtin_amdgcn_update_dpp(0, (int)v, 0x143, 0xc, 0xf, false));
  return v;
}

#define STAMP(i) do { } while (0)

__global__ __launch_bounds__(512, 2) void a2a_gemm_q8(Params P) {
  extern __shared__ __attribute__((aligned(16))) char smem[];
  LAS char* lds = (LAS char*)smem;
  const int tid = threadIdx.x;
  const int wid = __builtin_amdgcn_readfirstlane(tid >> 6);
  const int lane = tid & 63;
  const int b = blockIdx.x;
  const int xcd = b & 7, idx = b >> 3;
  const int mt = idx & 1, nt = xcd * 16 + (idx >> 1);
  const int rank = P.rank;
  char* const myws = P.ws[rank];
  unsigned call = 0;
  bool timed_out = false;

  if (wid >= 4) {
    const int r8 = lane >> 3, c8 = lane & 7;
    const unsigned va0 = (unsigned)(r8 * 1024 + ((c8 ^ ((0 + (lane >> 4)) & 7)) * 16));
    const unsigned va1 = (unsigned)(r8 * 1024 + ((c8 ^ ((4 + (lane >> 4)) & 7)) * 16));
    const int g4 = lane >> 4, c16 = lane & 15;
    const unsigned vw0 = (unsigned)(g4 * 32768 + ((c16 ^ ((g4 << 2) | 0)) * 16));
    const unsigned vw1 = (unsigned)(g4 * 32768 + ((c16 ^ ((g4 << 2) | 2)) * 16));
    auto wrsrc = __builtin_amdgcn_make_buffer_rsrc((void*)(P.W + nt * 256), 0, (int)((size_t)KTOT * NTOT * 2 - (size_t)nt * 256), 0x00020000);
    const char* const xloc = P.x + (size_t)rank * CHUNK_BYTES;
    auto issueAh = [&](int j, int h) {
      int p, s; kt_decode(j, rank, p, s);
      const char* base = (j < 8) ? xloc : (myws + WS_STG + (size_t)p * CHUNK_BYTES);
      auto rsrc = __builtin_amdgcn_make_buffer_rsrc((void*)base, 0, (int)CHUNK_BYTES, 0x00020000);
      LAS char* dst = lds + LDS_A + (j % NSA) * A_STAGE + h * 16384;
      const unsigned so = (unsigned)((mt * 256 + h * 128) * 1024) + s * 128;
#pragma unroll
      for (int i = 0; i < 16; ++i)
        __builtin_amdgcn_raw_ptr_buffer_load_lds(rsrc, (LAS void*)(dst + i * 1024), 16, (i & 1) ? va1 : va0, so + i * 8192, 0, 17  );
    };
    auto issueWh = [&](int j, int h) {
      int p, s; kt_decode(j, rank, p, s);
      const unsigned so = (unsigned)(p * 512 + s * 64) * 32768u + (unsigned)h * (8u * 131072u);
      LAS char* dst = lds + LDS_B + (j % NSB) * B_STAGE + h * 8192;
#pragma unroll
      for (int i = 0; i < 8; ++i)
        __builtin_amdgcn_raw_ptr_buffer_load_lds(wrsrc, (LAS void*)(dst + i * 1024), 16, (i & 2) ? vw1 : vw0, so + i * 131072, 0, 0);
    };
    if (wid < 6) {
      const int h = wid - 4;
      auto issueA = [&](int j) { issueAh(j, h); };
      auto issueW = [&](int j) { issueWh(j, h); };
      issueA(0); issueW(0); issueW(1);
      WAIT_VM(8);
      BAR_ONLY();
      WAIT_VM(0); BAR_ONLY();
      issueA(2); issueW(3);
      for (int t = 0; t < NKT; ++t) {
        if (t + 3 < NKT) WAIT_VM(24); else if (t + 2 < NKT) WAIT_VM(16); else WAIT_VM(0);
        BAR_ONLY();
        if (t + 3 < NKT) issueA(t + 3);
        if (t + 4 < NKT) issueW(t + 4);
      }
    } else if (wid == 6) {
      const bool active = b < 7 * 32;
      const int pj = b % 7, pu = b / 7;
      const int dest = (rank + 1 + pj) & 7;
      u32x4 v[16];
      const char* src = P.x + (size_t)dest * CHUNK_BYTES + (size_t)((pu & 31) * 16 + (lane >> 3)) * 1024 + (lane & 7) * 16;
      v[0] = *(const u32x4*)(src); v[1] = *(const u32x4*)(src + 8192);
      const unsigned callv = ld_agent32(myws + WS_CC + b * 4);
      asm volatile("" ::: "memory");
      issueWh(2, 0); issueWh(2, 1);
      asm volatile("" ::: "memory");
      const unsigned pcall = __builtin_amdgcn_readfirstlane(callv) + 1u;
      STAMP(0);
      auto drsrc = __builtin_amdgcn_make_buffer_rsrc((void*)(P.ws[dest] + WS_STG + (size_t)rank * CHUNK_BYTES), 0, (int)CHUNK_BYTES, 0x00020000);
      const unsigned voff = (unsigned)(((pu & 31) * 16 + (lane >> 3)) * 1024 + (lane & 7) * 16);
      char* fl = P.ws[dest] + WS_XF + ((size_t)((lane & (XF_REPS - 1)) * 2 + ((pu >> 4) & 1)) * NR + rank) * 128 + (pu & 15) * 4;
      auto push = [&](auto lo, auto hi) {
#pragma unroll
        for (int i = decltype(lo)::value; i < decltype(hi)::value; ++i)
          __builtin_amdgcn_raw_buffer_store_b128(v[i], drsrc, voff + (i & 1) * 8192 + (i >> 1) * 128, 0, 17  );
      };
      auto flag = [&](unsigned nslices, auto younger) {
        asm volatile("s_waitcnt vmcnt(%0)" :: "n"(decltype(younger)::value) : "memory");
        if (lane < XF_REPS) __hip_atomic_store((unsigned*)fl, pcall * 8u + nslices, __ATOMIC_RELAXED, __HIP_MEMORY_SCOPE_SYSTEM);
      };
      unsigned sink = 0;
      const char* const pfa = P.x + (size_t)rank * CHUNK_BYTES + (size_t)((mt * 256 + ((idx >> 1) & 15) * 16 + ((lane & 31) >> 1)) * 1024 + (lane & 1) * 64 + (lane >> 5) * 128);
      int t = 0;
      auto bars_until = [&](int T) { for (; t < T; ++t) BAR_ONLY(); };
      if (active) push(IC<0>{}, IC<2>{});
      STAMP(1);
      BAR_ONLY(); BAR_ONLY();
      if (active) WAIT_VM(2); else WAIT_VM(0);
#pragma unroll
      for (int i = 2; i < 16; ++i) v[i] = *(const u32x4*)(src + (i & 1) * 8192 + (i >> 1) * 128);
      asm volatile("" ::: "memory");
      asm volatile("global_load_dword %0, %1, off offset:384\n\tglobal_load_dword %0, %1, off offset:640\n\tglobal_load_dword %0, %1, off offset:768" : "+v"(sink) : "v"(pfa) : "memory");
      if (active) {
        const int ph = pu & (PUSH_STEP - 1);
        bars_until(PUSH_T0); flag(1, IC<14 + 3>{}); STAMP(2);
#define PUSH_SLICE(S) bars_until(PUSH_STEP * (S) - 2 + ph); push(IC<2 * (S)>{}, IC<2 * (S) + 2>{}); bars_until(PUSH_STEP * (S) - 2 + ph + PUSH_LAG); flag((S) + 1, IC<0>{});
        PUSH_SLICE(1) STAMP(3);
        PUSH_SLICE(2) PUSH_SLICE(3) PUSH_SLICE(4) PUSH_SLICE(5) PUSH_SLICE(6) PUSH_SLICE(7)
        STAMP(4);
#undef PUSH_SLICE
      }
      bars_until(NKT);
      asm volatile("s_waitcnt vmcnt(0)" :: "v"(sink) : "memory");
    } else {
      call = __builtin_amdgcn_readfirstlane(ld_agent32(myws + WS_CC + b * 4)) + 1u;
      const int pi = lane >> 3;
      const int src = (rank + 1 + pi) & 7;
      const char* fl = myws + WS_XF + ((size_t)(((idx >> 1) & (XF_REPS - 1)) * 2 + mt) * NR + src) * 128 + (lane & 7) * 8;
      auto flags_ok = [&](u64 v, unsigned need) { return pi == 7 || ((unsigned)v >= need && (unsigned)(v >> 32) >= need); };
      WAIT_VM(0);
      issueAh(1, 0); issueAh(1, 1);
      BAR_ONLY(); BAR_ONLY();
      WAIT_VM(0);
      u64 pre = 0;
      for (int t = 0; t < NKT; ++t) {
        const int j = t + 3 - 8;
        if (j >= 0 && j < 56 && j % 7 == 0) {
          const unsigned need = call * 8u + (unsigned)(j / 7) + 1u;
          if (!__all(flags_ok(pre, need))) {
            const u64 t0 = __builtin_amdgcn_s_memrealtime(); unsigned spins = 0;
            for (;;) {
              const u64 v = ld_sys64(fl);
              if (__all(flags_ok(v, need))) break;
              if (give_up(myws, call, t0, spins)) { timed_out = true; break; }
              __builtin_amdgcn_s_sleep(1);
            }
          }
          if (j == 0) STAMP(5); if (j == 7) STAMP(6); if (j == 49) STAMP(7);
        }
        if (j + 1 >= 0 && j + 1 < 56 && (j + 1) % 7 == 0 && !__all(flags_ok(pre, call * 8u + (unsigned)((j + 1) / 7) + 1u))) pre = ld_sys64(fl);
        BAR_ONLY();
      }
    }
  } else {
    const int wm = wid >> 1, wn = wid & 1;
    const int l15 = lane & 15, g = lane >> 4, q = l15 >> 2, pp = lane & 3;
    unsigned baseA[2], baseB[4];
#pragma unroll
    for (int ks = 0; ks < 2; ++ks) baseA[ks] = (unsigned)((wm * 128 + l15) * 128 + (((4 * ks + g) ^ (l15 >> 1)) * 16)) + LDS_A;
    const int swz = (q << 2) | ((g & 1) << 1);
#pragma unroll
    for (int c = 0; c < 4; ++c) baseB[c] = (unsigned)((8 * g + q) * 256 + (((8 * wn + 2 * c + (pp >> 1)) ^ swz) * 16) + 8 * (pp & 1)) + LDS_B;

    f32x4 acc[8][4];
#pragma unroll
    for (int m = 0; m < 8; ++m)
#pragma unroll
      for (int c = 0; c < 4; ++c) acc[m][c] = (f32x4){0.f, 0.f, 0.f, 0.f};

    const unsigned callv = ld_agent32(myws + WS_CC + b * 4);
    __builtin_amdgcn_s_setprio(1);
    if (wid == 0) STAMP(10);
    BAR_ONLY();
    if (wid == 0) STAMP(11);
    {
      bf16x8 bx[4][2], by[4][2], a0[2], a1[2];
      auto rdB1 = [&](bf16x8& dst, unsigned sb, int c, int ks) {
        const s16x4 lo = __builtin_amdgcn_ds_read_tr16_b64_v4i16((LAS s16x4*)(lds + (baseB[c] + sb) + ks * 8192));
        const s16x4 hi = __builtin_amdgcn_ds_read_tr16_b64_v4i16((LAS s16x4*)(lds + (baseB[c] + sb) + ks * 8192 + 1024));
        dst = __builtin_shufflevector(lo, hi, 0, 1, 2, 3, 4, 5, 6, 7);
      };
      auto rdA1 = [&](bf16x8 (&dst)[2], unsigned sa, int m) {
        dst[0] = *(const LAS bf16x8*)(lds + (baseA[0] + sa) + m * 2048);
        dst[1] = *(const LAS bf16x8*)(lds + (baseA[1] + sa) + m * 2048);
      };
#pragma unroll
      for (int c = 0; c < 4; ++c) { rdB1(bx[c][0], 0, c, 0); rdB1(bx[c][1], 0, c, 1); }
      rdA1(a0, 0, 0); rdA1(a1, 0, 1);
      BAR_LGKM();
      unsigned sa = 0, sbn = B_STAGE;
      auto tile = [&](bf16x8 (&bc)[4][2], bf16x8 (&bn)[4][2]) {
        unsigned san = sa + A_STAGE; if (san == NSA * A_STAGE) san = 0;
        bf16x8 a[10][2];
        a[0][0] = a0[0]; a[0][1] = a0[1]; a[1][0] = a1[0]; a[1][1] = a1[1];
#pragma unroll
        for (int m = 0; m < 8; ++m) {
#pragma unroll
          for (int i = 0; i < 4; ++i) {
            const int c = (m & 1) ? 3 - i : i;
            acc[m][c] = __builtin_amdgcn_mfma_f32_16x16x32_bf16(bc[c][0], a[m][0], acc[m][c], 0, 0, 0);
            if (i == 0) { if (m + 2 < 8) rdA1(a[m + 2], sa, m + 2); else rdA1(a[m + 2], san, m - 6); }
            if (i == 1 && m < 4) rdB1(bn[m][0], sbn, m, 0);
            if (i == 2 && m < 4) rdB1(bn[m][1], sbn, m, 1);
            acc[m][c] = __builtin_amdgcn_mfma_f32_16x16x32_bf16(bc[c][1], a[m][1], acc[m][c], 0, 0, 0);
            if (i == 0 || ((i == 1 || i == 2) && m < 4)) {
              __builtin_amdgcn_sched_group_barrier(0x8, 1, 0); __builtin_amdgcn_sched_group_barrier(0x100, 1, 0);
              __builtin_amdgcn_sched_group_barrier(0x8, 1, 0); __builtin_amdgcn_sched_group_barrier(0x100, 1, 0);
            }
            SCHED_FENCE();
          }
          if (m == 5) { BAR_LGKM(); SCHED_FENCE(); }
        }
        a0[0] = a[8][0]; a0[1] = a[8][1]; a1[0] = a[9][0]; a1[1] = a[9][1];
        sa = san;
        sbn += B_STAGE; if (sbn == NSB * B_STAGE) sbn = 0;
      };
      for (int t = 0; t < NKT; t += 2) { tile(bx, by); tile(by, bx); }
    }
    __builtin_amdgcn_s_setprio(0);
    if (wid == 0) STAMP(12);
    call = __builtin_amdgcn_readfirstlane(callv) + 1u;
    float am0 = 0.f, am1 = 0.f;
#pragma unroll
    for (int m = 0; m < 8; ++m)
#pragma unroll
      for (int c = 0; c < 4; ++c) {
        am0 = fmaxf(fmaxf(am0, fabsf(acc[m][c][0])), fabsf(acc[m][c][1]));
        am1 = fmaxf(fmaxf(am1, fabsf(acc[m][c][2])), fabsf(acc[m][c][3]));
      }
    const unsigned wmax = wave_umax(__float_as_uint(fmaxf(am0, am1)));
    if (lane == 63) __hip_atomic_store((u64*)(myws + WS_LG + (b * 4 + wid) * 8), ((u64)call << 32) | wmax, __ATOMIC_RELAXED, __HIP_MEMORY_SCOPE_AGENT);
    if (wid == 0) STAMP(8);
#pragma unroll
    for (int m = 0; m < 8; ++m)
#pragma unroll
      for (int c = 0; c < 4; ++c) {
        const int row = wm * 128 + m * 16 + l15;
        *(LAS f32x4*)(lds + row * 512 + (((wn * 16 + c * 4 + g) ^ (l15 & 7)) * 16)) = acc[m][c];
      }
  }

  BAR_LGKM();
  f32x4 yv[4][4];
  {
    const int r3 = lane >> 3, s8 = lane & 7;
#pragma unroll
    for (int i = 0; i < 4; ++i)
#pragma unroll
      for (int j = 0; j < 4; ++j)
        yv[i][j] = *(const LAS f32x4*)(lds + (wid * 32 + i * 8 + r3) * 512 + (((s8 * 4 + j) ^ r3) * 16));
  }
  if (wid == 7) {
    if (b == 0) {
      auto lgr = __builtin_amdgcn_make_buffer_rsrc((void*)(myws + WS_LG), 0, NWG * 4 * 8, 0x00020000);
      const u64 t0 = __builtin_amdgcn_s_memrealtime(); unsigned spins = 0;
      unsigned mx = 0;
      auto sweep = [&](u32x4 (&v)[8]) {
#pragma unroll
        for (int i = 0; i < 8; ++i) v[i] = __builtin_amdgcn_raw_buffer_load_b128(lgr, lane * 16 + i * 1024, 0, 16  );
        asm volatile("" ::: "memory");
      };
      auto check = [&](const u32x4 (&v)[8]) {
        bool ok = true; mx = 0;
#pragma unroll
        for (int i = 0; i < 8; ++i) { ok = ok && v[i][1] >= call && v[i][3] >= call; mx = max(mx, max(v[i][0], v[i][2])); }
        return __all(ok) != 0;
      };
      u32x4 va[8], vb[8];
      sweep(va);
      for (;;) {
        sweep(vb); if (check(va)) break;
        sweep(va); if (check(vb)) break;
        if (give_up(myws, call, t0, spins)) { timed_out = true; break; }
      }
      mx = __builtin_amdgcn_readlane(wave_umax(mx), 63);
      const u64 gr = ((u64)call << 32) | mx;
#pragma unroll
      for (int i = 0; i < FG_REPS / 8; ++i) st_sys64(P.ws[lane >> 3] + WS_FG + (i * 8 + (lane & 7)) * 128 + rank * 8, gr);
    }
    unsigned gmx, tm;
    {
      const char* fg = myws + WS_FG + (b & (FG_REPS - 1)) * 128 + (lane & 7) * 8;
      const u64 t0 = __builtin_amdgcn_s_memrealtime(); unsigned spins = 0;
      u64 v0 = ld_sys64(fg); unsigned m0 = ld_agent32(myws + WS_TMO);
      __builtin_amdgcn_s_sleep(3);
      u64 v1 = ld_sys64(fg); unsigned m1 = ld_agent32(myws + WS_TMO);
      __builtin_amdgcn_s_sleep(3);
      u64 v2 = ld_sys64(fg); unsigned m2 = ld_agent32(myws + WS_TMO);
      for (;;) {
        if (__all((unsigned)(v0 >> 32) >= call)) { gmx = (unsigned)v0; tm = m0; break; }
        v0 = ld_sys64(fg); m0 = ld_agent32(myws + WS_TMO);
        if (__all((unsigned)(v1 >> 32) >= call)) { gmx = (unsigned)v1; tm = m1; break; }
        v1 = ld_sys64(fg); m1 = ld_agent32(myws + WS_TMO);
        if (__all((unsigned)(v2 >> 32) >= call)) { gmx = (unsigned)v2; tm = m2; break; }
        v2 = ld_sys64(fg); m2 = ld_agent32(myws + WS_TMO);
        if (give_up(myws, call, t0, spins)) { gmx = 0; tm = call; break; }
      }
      gmx = __builtin_amdgcn_readlane(wave_umax(gmx), 63);
    }
    if (lane == 0) {
      LAS unsigned* o = (LAS unsigned*)(lds + LDS_MISC + 64);
      const float scale = __uint_as_float(gmx) / 127.0f;
      o[0] = __float_as_uint(1.0f / scale);
      o[1] = (tm == call || timed_out) ? 1u : 0u;
      st_agent32(myws + WS_CC + b * 4, call);
    }
    STAMP(9);
  }
  BAR_LGKM();
  const float inv = *(const LAS float*)(lds + LDS_MISC + 64);
  const unsigned bad = *(const LAS unsigned*)(lds + LDS_MISC + 68);
  if (wid == 0) STAMP(13);
  {
    auto orsrc = __builtin_amdgcn_make_buffer_rsrc((void*)P.out, 0, MLOC * NTOT, 0x00020000);
    const unsigned ovoff = (unsigned)((mt * 256 + wid * 32 + (lane >> 3)) * NTOT + nt * 128 + (lane & 7) * 16);
#pragma unroll
    for (int i = 0; i < 4; ++i) {
      unsigned d[4];
#pragma unroll
      for (int j = 0; j < 4; ++j) {
        unsigned w = 0;
#pragma unroll
        for (int r = 0; r < 4; ++r) {
          float u = __builtin_fmaf(yv[i][j][r], inv, 128.0f);
          u = __builtin_amdgcn_fmed3f(u, 1.0f, 255.0f);
          w = __builtin_amdgcn_cvt_pk_u8_f32(u, r, w);
        }
        d[j] = bad ? 0x80808080u : (w ^ 0x80808080u);
      }
      const u32x4 v = {d[0], d[1], d[2], d[3]};
      __builtin_amdgcn_raw_buffer_store_b128(v, orsrc, ovoff + i * 8 * NTOT, 0, OUT_AUX);
    }
  }
}

}

extern "C" void kernel_launch(int rank, int nranks, ncclComm_t comm,
                              hipStream_t stream,
                              void* const* d_in, const int* in_sizes, int n_in,
                              void* d_out, int out_size,
                              const int* in_dtypes, int out_dtype,
                              void* const* d_out_all,
                              void* const* d_ws_all, size_t ws_bytes) {
  (void)comm; (void)in_dtypes; (void)out_dtype; (void)d_out_all;
  if (nranks != NR || n_in != 2 || (unsigned)rank >= (unsigned)NR || ws_bytes < WS_END + 131072 ||
      in_sizes[0] != KTOT * 512 || in_sizes[1] != KTOT * NTOT || out_size != MLOC * NTOT) {
    fprintf(stderr, "unexpected problem shape\n");
    return;
  }
  HIP_CHECK(hipFuncSetAttribute((const void*)a2a_gemm_q8, hipFuncAttributeMaxDynamicSharedMemorySize, LDS_BYTES));
  Params P;
  P.x = (const char*)d_in[0];
  P.W = (const char*)d_in[1];
  P.out = (signed char*)d_out;
  for (int p = 0; p < NR; ++p) P.ws[p] = (char*)d_ws_all[p];
  P.rank = rank;
  hipLaunchKernelGGL(a2a_gemm_q8, dim3(NWG), dim3(512), LDS_BYTES, stream, P);
  HIP_CHECK(hipGetLastError());
}
```

```cpp
#include <hip/hip_runtime.h>
#include <rccl/rccl.h>
#include <cstdio>

#define HIP_CHECK(expr) do { hipError_t _e = (expr); if (_e != hipSuccess) { \
    fprintf(stderr, "HIP error %s at %s:%d\n", hipGetErrorString(_e), __FILE__, __LINE__); \
    return; } } while (0)

#define LAS __attribute__((address_space(3)))
typedef short bf16x8 __attribute__((ext_vector_type(8)));
typedef short s16x4 __attribute__((ext_vector_type(4)));
typedef float f32x4 __attribute__((ext_vector_type(4)));
typedef unsigned u32x4 __attribute__((ext_vector_type(4)));
typedef unsigned long long u64;

namespace {

constexpr int NR = 8;
constexpr int MLOC = 512, KTOT = 4096, NTOT = 16384;
constexpr int BM = 256, BN = 128, BK = 64, NKT = KTOT / BK;
constexpr int NWG = 256;
constexpr int A_STAGE = BM * BK * 2;
constexpr int B_STAGE = BK * BN * 2;
constexpr int NSA = 3, NSB = 3;
constexpr int LDS_A = 0, LDS_B = NSA * A_STAGE;
constexpr int LDS_BYTES = LDS_B + NSB * B_STAGE;
constexpr int LDS_MISC = 131072;
static_assert(LDS_BYTES <= 163840, "LDS");

constexpr size_t CHUNK_BYTES = (size_t)MLOC * 512 * 2;
constexpr size_t WS_STG = 0;
constexpr size_t WS_XF = WS_STG + NR * CHUNK_BYTES;
constexpr int XF_REPS = 4;
constexpr size_t WS_XF_BYTES = (size_t)XF_REPS * 2 * NR * 128;
constexpr size_t WS_LG = WS_XF + WS_XF_BYTES;
constexpr size_t WS_FG = WS_LG + NWG * 4 * 8;
constexpr int FG_REPS = 8;
constexpr size_t WS_CC = WS_FG + FG_REPS * 128;
constexpr size_t WS_TMO = WS_CC + NWG * 4;
constexpr size_t WS_END = WS_TMO + 128;

constexpr u64 DEADLINE_TICKS = 50000000ull;

struct Params {
  const char* x;
  const char* W;
  signed char* out;
  char* ws[NR];
  int rank;
};

template <int N> struct IC { static constexpr int value = N; };

#define BAR_LGKM() asm volatile("s_waitcnt lgkmcnt(0)\n\ts_barrier" ::: "memory")
#define BAR_ONLY() asm volatile("s_barrier" ::: "memory")
#define WAIT_VM(n) asm volatile("s_waitcnt vmcnt(" #n ")" ::: "memory")
#define SCHED_FENCE() __builtin_amdgcn_sched_barrier(0)

__device__ __forceinline__ void kt_decode(int j, int rank, int& p, int& s) {
  if (j < 8) { p = rank; s = j; }
  else { const int jj = j - 8; s = jj / 7; const int pi = jj - s * 7; p = (rank + 1 + pi) & 7; }
}
constexpr int PUSH_STEP = 4;
constexpr int PUSH_LAG = 3;
constexpr int PUSH_T0 = 1;
constexpr int OUT_AUX = 16;

__device__ __forceinline__ u64 ld_sys64(const char* p) { return __hip_atomic_load((const u64*)p, __ATOMIC_RELAXED, __HIP_MEMORY_SCOPE_SYSTEM); }
__device__ __forceinline__ void st_sys64(char* p, u64 v) { __hip_atomic_store((u64*)p, v, __ATOMIC_RELAXED, __HIP_MEMORY_SCOPE_SYSTEM); }
__device__ __forceinline__ unsigned ld_agent32(const char* p) { return __hip_atomic_load((const unsigned*)p, __ATOMIC_RELAXED, __HIP_MEMORY_SCOPE_AGENT); }
__device__ __forceinline__ void st_agent32(char* p, unsigned v) { __hip_atomic_store((unsigned*)p, v, __ATOMIC_RELAXED, __HIP_MEMORY_SCOPE_AGENT); }

__device__ __forceinline__ bool give_up(char* myws, unsigned call, u64 t0, unsigned& spins) {
  if ((++spins & 15u) != 0u) return false;
  const unsigned tm = __builtin_amdgcn_readfirstlane(ld_agent32(myws + WS_TMO));
  if (tm == call) return true;
  if (__builtin_amdgcn_s_memrealtime() - t0 > DEADLINE_TICKS) {
    st_agent32(myws + WS_TMO, call);
    asm volatile("s_waitcnt vmcnt(0)" ::: "memory");
    const u64 t1 = __builtin_amdgcn_s_memrealtime();
    while (__builtin_amdgcn_s_memrealtime() - t1 < 1000ull) __builtin_amdgcn_s_sleep(8);
    return true;
  }
  return false;
}

__device__ __forceinline__ unsigned wave_umax(unsigned v) {
  v = max(v, (unsigned)__builtin_amdgcn_update_dpp(0, (int)v, 0x111, 0xf, 0xf, false));
  v = max(v, (unsigned)__builtin_amdgcn_update_dpp(0, (int)v, 0x112, 0xf, 0xf, false));
  v = max(v, (unsigned)__builtin_amdgcn_update_dpp(0, (int)v, 0x114, 0xf, 0xf, false));
  v = max(v, (unsigned)__builtin_amdgcn_update_dpp(0, (int)v, 0x118, 0xf, 0xf, false));
  v = max(v, (unsigned)__builtin_amdgcn_update_dpp(0, (int)v, 0x142, 0xa, 0xf, false));
  v = max(v, (unsigned)__builtin_amdgcn_update_dpp(0, (int)v, 0x143, 0xc, 0xf, false));
  return v;
}


__global__ __launch_bounds__(512, 2) void a2a_gemm_q8(Params P) {
  extern __shared__ __attribute__((aligned(16))) char smem[];
  LAS char* lds = (LAS char*)smem;
  const int tid = threadIdx.x;
  const int wid = __builtin_amdgcn_readfirstlane(tid >> 6);
  const int lane = tid & 63;
  const int b = blockIdx.x;
  const int xcd = b & 7, idx = b >> 3;
  const int mt = idx & 1, nt = xcd * 16 + (idx >> 1);
  const int rank = P.rank;
  char* const myws = P.ws[rank];
  unsigned call = 0;
  bool timed_out = false;

  if (wid >= 4) {
    const int r8 = lane >> 3, c8 = lane & 7;
    const unsigned va0 = (unsigned)(r8 * 1024 + ((c8 ^ ((0 + (lane >> 4)) & 7)) * 16));
    const unsigned va1 = (unsigned)(r8 * 1024 + ((c8 ^ ((4 + (lane >> 4)) & 7)) * 16));
    const int g4 = lane >> 4, c16 = lane & 15;
    const unsigned vw0 = (unsigned)(g4 * 32768 + ((c16 ^ ((g4 << 2) | 0)) * 16));
    const unsigned vw1 = (unsigned)(g4 * 32768 + ((c16 ^ ((g4 << 2) | 2)) * 16));
    auto wrsrc = __builtin_amdgcn_make_buffer_rsrc((void*)(P.W + nt * 256), 0, (int)((size_t)KTOT * NTOT * 2 - (size_t)nt * 256), 0x00020000);
    const char* const xloc = P.x + (size_t)rank * CHUNK_BYTES;
    auto issueAh = [&](int j, int h) {
      int p, s; kt_decode(j, rank, p, s);
      const char* base = (j < 8) ? xloc : (myws + WS_STG + (size_t)p * CHUNK_BYTES);
      auto rsrc = __builtin_amdgcn_make_buffer_rsrc((void*)base, 0, (int)CHUNK_BYTES, 0x00020000);
      LAS char* dst = lds + LDS_A + (j % NSA) * A_STAGE + h * 16384;
      const unsigned so = (unsigned)((mt * 256 + h * 128) * 1024) + s * 128;
#pragma unroll
      for (int i = 0; i < 16; ++i)
        __builtin_amdgcn_raw_ptr_buffer_load_lds(rsrc, (LAS void*)(dst + i * 1024), 16, (i & 1) ? va1 : va0, so + i * 8192, 0, 17  );
    };
    auto issueWh = [&](int j, int h) {
      int p, s; kt_decode(j, rank, p, s);
      const unsigned so = (unsigned)(p * 512 + s * 64) * 32768u + (unsigned)h * (8u * 131072u);
      LAS char* dst = lds + LDS_B + (j % NSB) * B_STAGE + h * 8192;
#pragma unroll
      for (int i = 0; i < 8; ++i)
        __builtin_amdgcn_raw_ptr_buffer_load_lds(wrsrc, (LAS void*)(dst + i * 1024), 16, (i & 2) ? vw1 : vw0, so + i * 131072, 0, 0);
    };
    if (wid < 6) {
      const int h = wid - 4;
      auto issueA = [&](int j) { issueAh(j, h); };
      auto issueW = [&](int j) { issueWh(j, h); };
      issueA(0); issueW(0); issueW(1);
      WAIT_VM(8);
      BAR_ONLY();
      WAIT_VM(0); BAR_ONLY();
      issueA(2); issueW(3);
      for (int t = 0; t < NKT; ++t) {
        if (t + 3 < NKT) WAIT_VM(24); else if (t + 2 < NKT) WAIT_VM(16); else WAIT_VM(0);
        BAR_ONLY();
        if (t + 3 < NKT) issueA(t + 3);
        if (t + 4 < NKT) issueW(t + 4);
      }
    } else if (wid == 6) {
      const bool active = b < 7 * 32;
      const int pj = b % 7, pu = b / 7;
      const int dest = (rank + 1 + pj) & 7;
      u32x4 v[16];
      const char* src = P.x + (size_t)dest * CHUNK_BYTES + (size_t)((pu & 31) * 16 + (lane >> 3)) * 1024 + (lane & 7) * 16;
      v[0] = *(const u32x4*)(src); v[1] = *(const u32x4*)(src + 8192);
      const unsigned callv = ld_agent32(myws + WS_CC + b * 4);
      asm volatile("" ::: "memory");
      issueWh(2, 0); issueWh(2, 1);
      asm volatile("" ::: "memory");
      const unsigned pcall = __builtin_amdgcn_readfirstlane(callv) + 1u;
      auto drsrc = __builtin_amdgcn_make_buffer_rsrc((void*)(P.ws[dest] + WS_STG + (size_t)rank * CHUNK_BYTES), 0, (int)CHUNK_BYTES, 0x00020000);
      const unsigned voff = (unsigned)(((pu & 31) * 16 + (lane >> 3)) * 1024 + (lane & 7) * 16);
      char* fl = P.ws[dest] + WS_XF + ((size_t)((lane & (XF_REPS - 1)) * 2 + ((pu >> 4) & 1)) * NR + rank) * 128 + (pu & 15) * 4;
      auto push = [&](auto lo, auto hi) {
#pragma unroll
        for (int i = decltype(lo)::value; i < decltype(hi)::value; ++i)
          __builtin_amdgcn_raw_buffer_store_b128(v[i], drsrc, voff + (i & 1) * 8192 + (i >> 1) * 128, 0, 17  );
      };
      auto flag = [&](unsigned nslices, auto younger) {
        asm volatile("s_waitcnt vmcnt(%0)" :: "n"(decltype(younger)::value) : "memory");
        if (lane < XF_REPS) __hip_atomic_store((unsigned*)fl, pcall * 8u + nslices, __ATOMIC_RELAXED, __HIP_MEMORY_SCOPE_SYSTEM);
      };
      unsigned sink = 0;
      const char* const pfa = P.x + (size_t)rank * CHUNK_BYTES + (size_t)((mt * 256 + ((idx >> 1) & 15) * 16 + ((lane & 31) >> 1)) * 1024 + (lane & 1) * 64 + (lane >> 5) * 128);
      int t = 0;
      auto bars_until = [&](int T) { for (; t < T; ++t) BAR_ONLY(); };
      if (active) push(IC<0>{}, IC<2>{});
      BAR_ONLY(); BAR_ONLY();
      if (active) WAIT_VM(2); else WAIT_VM(0);
#pragma unroll
      for (int i = 2; i < 16; ++i) v[i] = *(const u32x4*)(src + (i & 1) * 8192 + (i >> 1) * 128);
      asm volatile("" ::: "memory");
      asm volatile("global_load_dword %0, %1, off offset:384\n\tglobal_load_dword %0, %1, off offset:640\n\tglobal_load_dword %0, %1, off offset:768" : "+v"(sink) : "v"(pfa) : "memory");
      if (active) {
        const int ph = pu & (PUSH_STEP - 1);
        bars_until(PUSH_T0); flag(1, IC<14 + 3>{});
#define PUSH_SLICE(S) bars_until(PUSH_STEP * (S) - 2 + ph); push(IC<2 * (S)>{}, IC<2 * (S) + 2>{}); bars_until(PUSH_STEP * (S) - 2 + ph + PUSH_LAG); flag((S) + 1, IC<0>{});
        PUSH_SLICE(1)
        PUSH_SLICE(2) PUSH_SLICE(3) PUSH_SLICE(4) PUSH_SLICE(5) PUSH_SLICE(6) PUSH_SLICE(7)
#undef PUSH_SLICE
      }
      bars_until(NKT);
      asm volatile("s_waitcnt vmcnt(0)" :: "v"(sink) : "memory");
    } else {
      call = __builtin_amdgcn_readfirstlane(ld_agent32(myws + WS_CC + b * 4)) + 1u;
      const int pi = lane >> 3;
      const int src = (rank + 1 + pi) & 7;
      const char* fl = myws + WS_XF + ((size_t)(((idx >> 1) & (XF_REPS - 1)) * 2 + mt) * NR + src) * 128 + (lane & 7) * 8;
      auto flags_ok = [&](u64 v, unsigned need) { return pi == 7 || ((unsigned)v >= need && (unsigned)(v >> 32) >= need); };
      WAIT_VM(0);
      issueAh(1, 0); issueAh(1, 1);
      BAR_ONLY(); BAR_ONLY();
      WAIT_VM(0);
      u64 pre = 0;
      for (int t = 0; t < NKT; ++t) {
        const int j = t + 3 - 8;
        if (j >= 0 && j < 56 && j % 7 == 0) {
          const unsigned need = call * 8u + (unsigned)(j / 7) + 1u;
          if (!__all(flags_ok(pre, need))) {
            const u64 t0 = __builtin_amdgcn_s_memrealtime(); unsigned spins = 0;
            for (;;) {
              const u64 v = ld_sys64(fl);
              if (__all(flags_ok(v, need))) { pre = v; break; }
              if (give_up(myws, call, t0, spins)) { timed_out = true; break; }
              __builtin_amdgcn_s_sleep(1);
            }
          }
        }
        if (j + 1 >= 0 && j + 1 < 56 && (j + 1) % 7 == 0 && !__all(flags_ok(pre, call * 8u + (unsigned)((j + 1) / 7) + 1u))) pre = ld_sys64(fl);
        BAR_ONLY();
      }
    }
  } else {
    const int wm = wid >> 1, wn = wid & 1;
    const int l15 = lane & 15, g = lane >> 4, q = l15 >> 2, pp = lane & 3;
    unsigned baseA[2], baseB[4];
#pragma unroll
    for (int ks = 0; ks < 2; ++ks) baseA[ks] = (unsigned)((wm * 128 + l15) * 128 + (((4 * ks + g) ^ (l15 >> 1)) * 16)) + LDS_A;
    const int swz = (q << 2) | ((g & 1) << 1);
#pragma unroll
    for (int c = 0; c < 4; ++c) baseB[c] = (unsigned)((8 * g + q) * 256 + (((8 * wn + 2 * c + (pp >> 1)) ^ swz) * 16) + 8 * (pp & 1)) + LDS_B;

    f32x4 acc[8][4];
#pragma unroll
    for (int m = 0; m < 8; ++m)
#pragma unroll
      for (int c = 0; c < 4; ++c) acc[m][c] = (f32x4){0.f, 0.f, 0.f, 0.f};

    const unsigned callv = ld_agent32(myws + WS_CC + b * 4);
    __builtin_amdgcn_s_setprio(1);
    BAR_ONLY();
    {
      bf16x8 bx[4][2], by[4][2], a0[2], a1[2];
      auto rdB1 = [&](bf16x8& dst, unsigned sb, int c, int ks) {
        const s16x4 lo = __builtin_amdgcn_ds_read_tr16_b64_v4i16((LAS s16x4*)(lds + (baseB[c] + sb) + ks * 8192));
        const s16x4 hi = __builtin_amdgcn_ds_read_tr16_b64_v4i16((LAS s16x4*)(lds + (baseB[c] + sb) + ks * 8192 + 1024));
        dst = __builtin_shufflevector(lo, hi, 0, 1, 2, 3, 4, 5, 6, 7);
      };
      auto rdA1 = [&](bf16x8 (&dst)[2], unsigned sa, int m) {
        dst[0] = *(const LAS bf16x8*)(lds + (baseA[0] + sa) + m * 2048);
        dst[1] = *(const LAS bf16x8*)(lds + (baseA[1] + sa) + m * 2048);
      };
#pragma unroll
      for (int c = 0; c < 4; ++c) { rdB1(bx[c][0], 0, c, 0); rdB1(bx[c][1], 0, c, 1); }
      rdA1(a0, 0, 0); rdA1(a1, 0, 1);
      BAR_LGKM();
      unsigned sa = 0, sbn = B_STAGE;
      auto tile = [&](bf16x8 (&bc)[4][2], bf16x8 (&bn)[4][2]) {
        unsigned san = sa + A_STAGE; if (san == NSA * A_STAGE) san = 0;
        bf16x8 a[10][2];
        a[0][0] = a0[0]; a[0][1] = a0[1]; a[1][0] = a1[0]; a[1][1] = a1[1];
#pragma unroll
        for (int m = 0; m < 8; ++m) {
#pragma unroll
          for (int i = 0; i < 4; ++i) {
            const int c = (m & 1) ? 3 - i : i;
            acc[m][c] = __builtin_amdgcn_mfma_f32_16x16x32_bf16(bc[c][0], a[m][0], acc[m][c], 0, 0, 0);
            if (i == 0) { if (m + 2 < 8) rdA1(a[m + 2], sa, m + 2); else rdA1(a[m + 2], san, m - 6); }
            if (i == 1 && m < 4) rdB1(bn[m][0], sbn, m, 0);
            if (i == 2 && m < 4) rdB1(bn[m][1], sbn, m, 1);
            acc[m][c] = __builtin_amdgcn_mfma_f32_16x16x32_bf16(bc[c][1], a[m][1], acc[m][c], 0, 0, 0);
            if (i == 0 || ((i == 1 || i == 2) && m < 4)) {
              __builtin_amdgcn_sched_group_barrier(0x8, 1, 0); __builtin_amdgcn_sched_group_barrier(0x100, 1, 0);
              __builtin_amdgcn_sched_group_barrier(0x8, 1, 0); __builtin_amdgcn_sched_group_barrier(0x100, 1, 0);
            }
            SCHED_FENCE();
          }
          if (m == 5) { BAR_LGKM(); SCHED_FENCE(); }
        }
        a0[0] = a[8][0]; a0[1] = a[8][1]; a1[0] = a[9][0]; a1[1] = a[9][1];
        sa = san;
        sbn += B_STAGE; if (sbn == NSB * B_STAGE) sbn = 0;
      };
      for (int t = 0; t < NKT; t += 2) { tile(bx, by); tile(by, bx); }
    }
    __builtin_amdgcn_s_setprio(0);
    call = __builtin_amdgcn_readfirstlane(callv) + 1u;
    float am0 = 0.f, am1 = 0.f;
#pragma unroll
    for (int m = 0; m < 8; ++m)
#pragma unroll
      for (int c = 0; c < 4; ++c) {
        am0 = fmaxf(fmaxf(am0, fabsf(acc[m][c][0])), fabsf(acc[m][c][1]));
        am1 = fmaxf(fmaxf(am1, fabsf(acc[m][c][2])), fabsf(acc[m][c][3]));
      }
    const unsigned wmax = wave_umax(__float_as_uint(fmaxf(am0, am1)));
    if (lane == 63) __hip_atomic_store((u64*)(myws + WS_LG + (b * 4 + wid) * 8), ((u64)call << 32) | wmax, __ATOMIC_RELAXED, __HIP_MEMORY_SCOPE_AGENT);
#pragma unroll
    for (int m = 0; m < 8; ++m)
#pragma unroll
      for (int c = 0; c < 4; ++c) {
        const int row = wm * 128 + m * 16 + l15;
        *(LAS f32x4*)(lds + row * 512 + (((wn * 16 + c * 4 + g) ^ (l15 & 7)) * 16)) = acc[m][c];
      }
  }

  BAR_LGKM();
  f32x4 yv[4][4];
  {
    const int r3 = lane >> 3, s8 = lane & 7;
#pragma unroll
    for (int i = 0; i < 4; ++i)
#pragma unroll
      for (int j = 0; j < 4; ++j)
        yv[i][j] = *(const LAS f32x4*)(lds + (wid * 32 + i * 8 + r3) * 512 + (((s8 * 4 + j) ^ r3) * 16));
  }
  if (wid == 7) {
    if (b == 0) {
      auto lgr = __builtin_amdgcn_make_buffer_rsrc((void*)(myws + WS_LG), 0, NWG * 4 * 8, 0x00020000);
      const u64 t0 = __builtin_amdgcn_s_memrealtime(); unsigned spins = 0;
      unsigned mx = 0;
      auto sweep = [&](u32x4 (&v)[8]) {
#pragma unroll
        for (int i = 0; i < 8; ++i) v[i] = __builtin_amdgcn_raw_buffer_load_b128(lgr, lane * 16 + i * 1024, 0, 16  );
        asm volatile("" ::: "memory");
      };
      auto check = [&](const u32x4 (&v)[8]) {
        bool ok = true; mx = 0;
#pragma unroll
        for (int i = 0; i < 8; ++i) { ok = ok && v[i][1] >= call && v[i][3] >= call; mx = max(mx, max(v[i][0], v[i][2])); }
        return __all(ok) != 0;
      };
      u32x4 va[8], vb[8];
      sweep(va);
      for (;;) {
        sweep(vb); if (check(va)) break;
        sweep(va); if (check(vb)) break;
        if (give_up(myws, call, t0, spins)) { timed_out = true; break; }
      }
      mx = __builtin_amdgcn_readlane(wave_umax(mx), 63);
      const u64 gr = ((u64)call << 32) | mx;
#pragma unroll
      for (int i = 0; i < FG_REPS / 8; ++i) st_sys64(P.ws[lane >> 3] + WS_FG + (i * 8 + (lane & 7)) * 128 + rank * 8, gr);
    }
    unsigned gmx, tm;
    {
      const char* fg = myws + WS_FG + (b & (FG_REPS - 1)) * 128 + (lane & 7) * 8;
      const u64 t0 = __builtin_amdgcn_s_memrealtime(); unsigned spins = 0;
      u64 v0 = ld_sys64(fg); unsigned m0 = ld_agent32(myws + WS_TMO);
      __builtin_amdgcn_s_sleep(3);
      u64 v1 = ld_sys64(fg); unsigned m1 = ld_agent32(myws + WS_TMO);
      __builtin_amdgcn_s_sleep(3);
      u64 v2 = ld_sys64(fg); unsigned m2 = ld_agent32(myws + WS_TMO);
      for (;;) {
        if (__all((unsigned)(v0 >> 32) >= call)) { gmx = (unsigned)v0; tm = m0; break; }
        v0 = ld_sys64(fg); m0 = ld_agent32(myws + WS_TMO);
        if (__all((unsigned)(v1 >> 32) >= call)) { gmx = (unsigned)v1; tm = m1; break; }
        v1 = ld_sys64(fg); m1 = ld_agent32(myws + WS_TMO);
        if (__all((unsigned)(v2 >> 32) >= call)) { gmx = (unsigned)v2; tm = m2; break; }
        v2 = ld_sys64(fg); m2 = ld_agent32(myws + WS_TMO);
        if (give_up(myws, call, t0, spins)) { gmx = 0; tm = call; break; }
      }
      gmx = __builtin_amdgcn_readlane(wave_umax(gmx), 63);
    }
    if (lane == 0) {
      LAS unsigned* o = (LAS unsigned*)(lds + LDS_MISC + 64);
      const float scale = __uint_as_float(gmx) / 127.0f;
      o[0] = __float_as_uint(1.0f / scale);
      o[1] = (tm == call || timed_out) ? 1u : 0u;
      st_agent32(myws + WS_CC + b * 4, call);
    }
  }
  BAR_LGKM();
  const float inv = *(const LAS float*)(lds + LDS_MISC + 64);
  const unsigned bad = *(const LAS unsigned*)(lds + LDS_MISC + 68);
  {
    auto orsrc = __builtin_amdgcn_make_buffer_rsrc((void*)P.out, 0, MLOC * NTOT, 0x00020000);
    const unsigned ovoff = (unsigned)((mt * 256 + wid * 32 + (lane >> 3)) * NTOT + nt * 128 + (lane & 7) * 16);
#pragma unroll
    for (int i = 0; i < 4; ++i) {
      unsigned d[4];
#pragma unroll
      for (int j = 0; j < 4; ++j) {
        unsigned w = 0;
#pragma unroll
        for (int r = 0; r < 4; ++r) {
          float u = __builtin_fmaf(yv[i][j][r], inv, 128.0f);
          u = __builtin_amdgcn_fmed3f(u, 1.0f, 255.0f);
          w = __builtin_amdgcn_cvt_pk_u8_f32(u, r, w);
        }
        d[j] = bad ? 0x80808080u : (w ^ 0x80808080u);
      }
      const u32x4 v = {d[0], d[1], d[2], d[3]};
      __builtin_amdgcn_raw_buffer_store_b128(v, orsrc, ovoff + i * 8 * NTOT, 0, OUT_AUX);
    }
  }
}

}

extern "C" void kernel_launch(int rank, int nranks, ncclComm_t comm,
                              hipStream_t stream,
                              void* const* d_in, const int* in_sizes, int n_in,
                              void* d_out, int out_size,
                              const int* in_dtypes, int out_dtype,
                              void* const* d_out_all,
                              void* const* d_ws_all, size_t ws_bytes) {
  (void)comm; (void)in_dtypes; (void)out_dtype; (void)d_out_all;
  if (nranks != NR || n_in != 2 || (unsigned)rank >= (unsigned)NR || ws_bytes < WS_END ||
      in_sizes[0] != KTOT * 512 || in_sizes[1] != KTOT * NTOT || out_size != MLOC * NTOT) {
    fprintf(stderr, "unexpected problem shape\n");
    return;
  }
  HIP_CHECK(hipFuncSetAttribute((const void*)a2a_gemm_q8, hipFuncAttributeMaxDynamicSharedMemorySize, LDS_BYTES));
  Params P;
  P.x = (const char*)d_in[0];
  P.W = (const char*)d_in[1];
  P.out = (signed char*)d_out;
  for (int p = 0; p < NR; ++p) P.ws[p] = (char*)d_ws_all[p];
  P.rank = rank;
  hipLaunchKernelGGL(a2a_gemm_q8, dim3(NWG), dim3(512), LDS_BYTES, stream, P);
  HIP_CHECK(hipGetLastError());
}
```
